```python
import numpy as np

kernel_name = "gpu_gemm_a2a_m4096_k4096_n16384_f32_reluqe4m3"


ABSMAX_RTOL = 0.09

COMPUTE_DTYPE = "float32"
EPILOGUE = "relu_quant_e4m3"
IN_QUANT = None
OUT_QUANT = "e4m3"
IN_DTYPES = {"x": "float32", "w_mat": "float32"}
WORLD_SIZE = 8

def setup_inputs():

    x = np.random.randn(4096, 4096).astype(np.float32)
    w_mat = (np.random.randn(4096, 16384) / np.sqrt(4096)).astype(np.float32)
    return {"x": x, "w_mat": w_mat}

def _snap_fp8(x, mant_bits, max_abs, min_normal):
    a = np.abs(x.astype(np.float64))
    _m, _e = np.frexp(a)
    q = np.where(a >= min_normal, np.ldexp(1.0, _e - 1 - mant_bits),
                 min_normal * 2.0 ** -mant_bits)
    snapped = np.minimum(np.round(a / q) * q, max_abs)
    return (np.sign(x) * snapped).astype(np.float32)
def _quant_dequant(y, amax):
    scale = amax / 448.0
    q = _snap_fp8(y / scale, 3, 448.0, 2.0 ** -6)
    return (q * scale).astype(np.float32)
def _epilogue(y):
    y = np.maximum(y, 0.0)
    return _quant_dequant(y, np.abs(y).max())

def reference(x, w_mat):
    y = (x @ w_mat).astype(np.float32)
    return _epilogue(y)
```

```cpp
#include <hip/hip_runtime.h>
#include <rccl/rccl.h>
#include <cstdio>
#include <cstdint>

#define HIP_CHECK(expr) do { hipError_t _e = (expr); if (_e != hipSuccess) { \
    fprintf(stderr, "HIP error %s at %s:%d\n", hipGetErrorString(_e), __FILE__, __LINE__); \
    return; } } while (0)

#define GAS __attribute__((address_space(1)))
#define LAS __attribute__((address_space(3)))
typedef float f32x4 __attribute__((ext_vector_type(4)));
typedef float f32x2 __attribute__((ext_vector_type(2)));
typedef unsigned v4u __attribute__((ext_vector_type(4)));
typedef __bf16 bf16x8 __attribute__((ext_vector_type(8)));
typedef __bf16 bf16x2 __attribute__((ext_vector_type(2)));
typedef unsigned long long u64;

constexpr int NR = 8;
constexpr int MLOC = 512, KDIM = 4096, NDIM = 16384, NLOC = 2048;
constexpr int BM = 256, BN = 128, KH = 32;
constexpr int NH = KDIM / KH;
constexpr int NWG = 256;
constexpr int NTHREADS = 512;
constexpr int NA = 8, A_SLOT = 16384;
constexpr int NB = 3, B_SLOT = 8192;
constexpr int LDS_A = 0, LDS_B = NA * A_SLOT, LDS_CTL = LDS_B + NB * B_SLOT;
constexpr int CTL_FULL = LDS_CTL, CTL_FREE = LDS_CTL + 16, CTL_WMAX = LDS_CTL + 32, CTL_GMAX = LDS_CTL + 48,
              CTL_BAD = LDS_CTL + 64;
constexpr int LDS_BYTES = LDS_CTL + 128;
constexpr int WD = 6;
constexpr size_t WS_CALL = 0, WS_TMO = 128;
constexpr size_t WS_AMAX = 4096;
constexpr size_t WS_DONE = 65536;
constexpr size_t WS_XB = 1u << 20;
constexpr size_t WS_END = WS_XB + (size_t)MLOC * KDIM * 2;
constexpr u64 DEADLINE = 50000000ull;

struct Params {
    const float* x;
    const float* w;
    unsigned char* out[NR];
    unsigned char* ws[NR];
    int rank;
    int phase;
};

__device__ __forceinline__ u64 rt() {
    u64 t; asm volatile("s_memrealtime %0\n\ts_waitcnt lgkmcnt(0)" : "=s"(t) : : "memory"); return t;
}
__device__ __forceinline__ unsigned cvt_bf16x2(float lo, float hi) {
    f32x2 v = {lo, hi};
    return __builtin_bit_cast(unsigned, __builtin_convertvector(v, bf16x2));
}
__device__ __forceinline__ void lds_st_u32(unsigned addr, unsigned v) {
    asm volatile("ds_write_b32 %0, %1" : : "v"(addr), "v"(v) : "memory");
}
__device__ __forceinline__ unsigned lds_min4(unsigned addr) {
    v4u v;
    asm volatile("ds_read_b128 %0, %1\n\ts_waitcnt lgkmcnt(0)" : "=&v"(v) : "v"(addr) : "memory");
    unsigned m = min(min(v.x, v.y), min(v.z, v.w));
    return __builtin_amdgcn_readfirstlane(m);
}
__device__ __forceinline__ unsigned lds_wait_min4(unsigned addr, unsigned need, int& dead) {
    unsigned m = lds_min4(addr);
    if (m < need && !dead) {
        const u64 t0 = rt();
        unsigned sp = 0;
        while (m < need) {
            __builtin_amdgcn_s_sleep(1);
            m = lds_min4(addr);
            if ((++sp & 1023u) == 0u && rt() - t0 > DEADLINE) { dead = 1; break; }
        }
    }
    return m;
}
__device__ __forceinline__ float wave_max(float v) {
#pragma unroll
    for (int o = 32; o >= 1; o >>= 1) v = fmaxf(v, __shfl_xor(v, o));
    return v;
}

__global__ void __launch_bounds__(NTHREADS, 2) fused_kernel(Params p) {
    extern __shared__ __attribute__((aligned(16))) unsigned char lds_raw[];
    LAS unsigned char* lds = (LAS unsigned char*)lds_raw;
    const int tid = threadIdx.x;
    const int wave = __builtin_amdgcn_readfirstlane(tid >> 6);
    const int lane = tid & 63;
    const int b = blockIdx.x;
    unsigned char* myws = p.ws[p.rank];

    if (p.phase == 0) {
        const int mt = b & 1, h = b >> 1;
#pragma unroll
        for (int i = 0; i < 2; ++i) {
            const int mf = wave * 2 + i;
            const int row = mt * BM + mf * 16 + (lane & 15);
            const GAS f32x4* src = (const GAS f32x4*)(p.x + (size_t)row * KDIM + h * KH + (lane >> 4) * 8);
            const f32x4 v0 = src[0], v1 = src[1];
            v4u o;
            o.x = cvt_bf16x2(v0.x, v0.y); o.y = cvt_bf16x2(v0.z, v0.w);
            o.z = cvt_bf16x2(v1.x, v1.y); o.w = cvt_bf16x2(v1.z, v1.w);
            *(GAS v4u*)(myws + WS_XB + ((size_t)(mt * NH + h) * 16 + mf) * 1024 + lane * 16) = o;
        }
        return;
    }

    const int grp = b & 7, idx = b >> 3;
    const int mt = idx & 1, ntl = idx >> 1;
    const int dest = grp;
    const int n0 = dest * NLOC + ntl * BN;

    const unsigned call = __hip_atomic_load((GAS unsigned*)(myws + WS_CALL), __ATOMIC_RELAXED, __HIP_MEMORY_SCOPE_AGENT) + 1u;

    if (tid < 32) ((LAS unsigned*)(lds + LDS_CTL))[tid] = (tid == 3) ? 0xffffffffu : 0u;
    __syncthreads();

    f32x4 acc[8][4];
    int dead = 0;

    if (wave < 4) {
        const int wm = wave & 1, wn = wave >> 1;
        const unsigned abase = LDS_A + wm * 8192 + lane * 16;
        const unsigned bbase = LDS_B + wn * 4096 + lane * 16;
        const unsigned free_addr = CTL_FREE + wave * 4;
#pragma unroll
        for (int i = 0; i < 8; ++i)
#pragma unroll
            for (int c = 0; c < 4; ++c) acc[i][c] = (f32x4){0.f, 0.f, 0.f, 0.f};
        v4u fa0[8], fb0[4], fa1[8], fb1[4];
        v4u fl;
        int sa = 0, sb = 0;
        lds_wait_min4(CTL_FULL, 1u, dead);
#define LOADSET(FA, FB) do { \
        _Pragma("unroll") for (int c = 0; c < 4; ++c) FB[c] = *(const LAS v4u*)(lds + bbase + sb + c * 1024); \
        _Pragma("unroll") for (int i = 0; i < 8; ++i) FA[i] = *(const LAS v4u*)(lds + abase + sa + i * 1024); \
        fl = *(const volatile LAS v4u*)(lds + CTL_FULL); \
        sa = (sa + A_SLOT) & (NA * A_SLOT - 1); sb += B_SLOT; if (sb == NB * B_SLOT) sb = 0; } while (0)
#define MMASET(FA, FB) do { \
        _Pragma("unroll") for (int i = 0; i < 8; ++i) \
        _Pragma("unroll") for (int c = 0; c < 4; ++c) \
            acc[i][c] = __builtin_amdgcn_mfma_f32_16x16x32_bf16(__builtin_bit_cast(bf16x8, FB[c]), \
                            __builtin_bit_cast(bf16x8, FA[i]), acc[i][c], 0, 0, 0); } while (0)
#define SCHEDSET() do { \
        _Pragma("unroll") for (int s = 0; s < 13; ++s) { \
            __builtin_amdgcn_sched_group_barrier(0x008, 2, 0); \
            __builtin_amdgcn_sched_group_barrier(0x100, 1, 0); } \
        __builtin_amdgcn_sched_group_barrier(0x008, 6, 0); } while (0)
#define PHASETOP(H) do { \
        asm volatile("s_waitcnt lgkmcnt(0)" ::: "memory"); \
        lds_st_u32(free_addr, (unsigned)(H) + 1u); \
        unsigned av = __builtin_amdgcn_readfirstlane(min(min(fl.x, fl.y), fl.z)); \
        if (__builtin_expect(av < (unsigned)(H) + 2u, 0)) lds_wait_min4(CTL_FULL, (unsigned)(H) + 2u, dead); \
        __builtin_amdgcn_sched_barrier(0); } while (0)
        LOADSET(fa0, fb0);
        for (int h = 0; h < NH - 2; h += 2) {
            PHASETOP(h);
            LOADSET(fa1, fb1); MMASET(fa0, fb0); SCHEDSET();
            __builtin_amdgcn_sched_barrier(0);
            PHASETOP(h + 1);
            LOADSET(fa0, fb0); MMASET(fa1, fb1); SCHEDSET();
            __builtin_amdgcn_sched_barrier(0);
        }
        PHASETOP(NH - 2);
        LOADSET(fa1, fb1); MMASET(fa0, fb0); SCHEDSET();
        __builtin_amdgcn_sched_barrier(0);
        MMASET(fa1, fb1);
        float m = 0.f;
#pragma unroll
        for (int i = 0; i < 8; ++i)
#pragma unroll
            for (int c = 0; c < 4; ++c)
                m = fmaxf(fmaxf(m, fmaxf(acc[i][c].x, acc[i][c].y)), fmaxf(acc[i][c].z, acc[i][c].w));
        m = wave_max(m);
        if (lane == 0) *(volatile LAS float*)(lds + CTL_WMAX + wave * 4) = m;
    } else if (wave == 4) {
        const unsigned v0 = lane * 16, v1 = v0 + 4096, v2 = v0 + 8192, v3 = v0 + 12288;
        const unsigned char* src = myws + WS_XB + (size_t)mt * NH * A_SLOT;
        unsigned freed = 0;
        int pub = 0;
        for (int h = 0; h < NH; ++h) {
            if (h >= NA && freed < (unsigned)(h - NA + 1)) freed = lds_wait_min4(CTL_FREE, (unsigned)(h - NA + 1), dead);
            const unsigned dst = LDS_A + (h & (NA - 1)) * A_SLOT;
            const unsigned char* s0 = src + (size_t)h * A_SLOT;
            unsigned keep;
#define DMA4(V) "global_load_lds_dwordx4 " V ", %5 offset:0\n\t" \
                "global_load_lds_dwordx4 " V ", %5 offset:1024\n\t" \
                "global_load_lds_dwordx4 " V ", %5 offset:2048\n\t" \
                "global_load_lds_dwordx4 " V ", %5 offset:3072\n\t"
            asm volatile(
                "s_mov_b32 %0, m0\n\t"
                "s_mov_b32 m0, %6\n\t"
                "s_nop 0\n\t"
                DMA4("%1")
                "s_add_u32 m0, m0, 0x1000\n\t"
                "s_nop 0\n\t"
                DMA4("%2")
                "s_add_u32 m0, m0, 0x1000\n\t"
                "s_nop 0\n\t"
                DMA4("%3")
                "s_add_u32 m0, m0, 0x1000\n\t"
                "s_nop 0\n\t"
                DMA4("%4")
                "s_mov_b32 m0, %0"
                : "=&s"(keep) : "v"(v0), "v"(v1), "v"(v2), "v"(v3), "s"(s0), "s"(dst) : "memory", "scc");
            if (h >= 2) {
                asm volatile("s_waitcnt vmcnt(32)" ::: "memory");
                ++pub; lds_st_u32(CTL_FULL, (unsigned)pub);
            }
        }
        asm volatile("s_waitcnt vmcnt(16)" ::: "memory");
        lds_st_u32(CTL_FULL, (unsigned)(NH - 1));
        asm volatile("s_waitcnt vmcnt(0)" ::: "memory");
        lds_st_u32(CTL_FULL, (unsigned)NH);
    } else if (wave < 7) {
        const int ww = wave - 5;
        const __amdgpu_buffer_rsrc_t rs = __builtin_amdgcn_make_buffer_rsrc((void*)p.w, 0, (int)((size_t)KDIM * NDIM * 4), 0x00020000);
        const unsigned voff = (unsigned)(lane >> 4) * 8u * (NDIM * 4) + (unsigned)(lane & 15) * 16u;
        const unsigned cb = (unsigned)(n0 + ww * 64) * 4u;
        const unsigned wbase = LDS_B + ww * 4096 + lane * 16;
        const unsigned full_addr = CTL_FULL + 4 + ww * 4;
        f32x4 r[WD][8];
#pragma unroll
        for (int d = 0; d < WD; ++d)
#pragma unroll
            for (int j = 0; j < 8; ++j)
                r[d][j] = __builtin_bit_cast(f32x4, __builtin_amdgcn_raw_buffer_load_b128(rs, voff, cb + (unsigned)(d * KH + j) * (NDIM * 4), 0));
        unsigned freed = 0;
        int sb = 0;
        for (int h0 = 0; h0 < NH; h0 += WD) {
#pragma unroll
            for (int d = 0; d < WD; ++d) {
                const int h = h0 + d;
                v4u f[4];
#pragma unroll
                for (int c = 0; c < 4; ++c) {
                    f[c].x = cvt_bf16x2(r[d][0][c], r[d][1][c]);
                    f[c].y = cvt_bf16x2(r[d][2][c], r[d][3][c]);
                    f[c].z = cvt_bf16x2(r[d][4][c], r[d][5][c]);
                    f[c].w = cvt_bf16x2(r[d][6][c], r[d][7][c]);
                }
                asm volatile("" : "+v"(f[0]), "+v"(f[1]), "+v"(f[2]), "+v"(f[3]));
#pragma unroll
                for (int j = 0; j < 8; ++j)
                    r[d][j] = __builtin_bit_cast(f32x4, __builtin_amdgcn_raw_buffer_load_b128(rs, voff, cb + (unsigned)((h + WD) * KH + j) * (NDIM * 4), 0));
                if (h < NH) {
                    if (h >= NB && freed < (unsigned)(h - NB + 1)) freed = lds_wait_min4(CTL_FREE, (unsigned)(h - NB + 1), dead);
#pragma unroll
                    for (int c = 0; c < 4; ++c) *(LAS v4u*)(lds + wbase + sb + c * 1024) = f[c];
                    asm volatile("s_waitcnt lgkmcnt(0)" ::: "memory");
                    lds_st_u32(full_addr, (unsigned)(h + 1));
                    sb += B_SLOT; if (sb == NB * B_SLOT) sb = 0;
                }
            }
        }
    }
    asm volatile("s_waitcnt lgkmcnt(0)" ::: "memory");
    __builtin_amdgcn_s_barrier();

    const size_t tab = WS_AMAX + (size_t)(call & 1u) * (NR * NWG * 8);
    GAS unsigned* tmo = (GAS unsigned*)(myws + WS_TMO);
    if (wave == 7 && lane < NR) {
        const v4u wm4 = *(const volatile LAS v4u*)(lds + CTL_WMAX);
        const unsigned mb = max(max(wm4.x, wm4.y), max(wm4.z, wm4.w));
        GAS u64* g = (GAS u64*)(p.ws[lane] + tab + ((size_t)p.rank * NWG + b) * 8);
        __hip_atomic_store(g, ((u64)call << 32) | mb, __ATOMIC_RELAXED, __HIP_MEMORY_SCOPE_SYSTEM);
    }
    if (wave >= 4) {
        const __amdgpu_buffer_rsrc_t rs = __builtin_amdgcn_make_buffer_rsrc((void*)(myws + tab), 0, NR * NWG * 8, 0x00020000);
        const unsigned voff = (unsigned)(wave - 4) * 4096u + lane * 16u;
        const u64 t0 = rt();
        unsigned gm = 0, bad = 0;
        for (unsigned sp = 0;; ++sp) {
            asm volatile("" ::: "memory");
            v4u g[4];
#pragma unroll
            for (int i = 0; i < 4; ++i) g[i] = __builtin_amdgcn_raw_buffer_load_b128(rs, voff + i * 1024, 0, 17);
            bool ok = true; gm = 0;
#pragma unroll
            for (int i = 0; i < 4; ++i) { ok = ok && g[i].y >= call && g[i].w >= call; gm = max(gm, max(g[i].x, g[i].z)); }
            if (__all(ok)) break;
            if ((sp & 63u) == 63u) {
                if (__builtin_amdgcn_readfirstlane(__hip_atomic_load(tmo, __ATOMIC_RELAXED, __HIP_MEMORY_SCOPE_AGENT)) == call || rt() - t0 > DEADLINE) {
                    if (lane == 0) __hip_atomic_store(tmo, call, __ATOMIC_RELAXED, __HIP_MEMORY_SCOPE_AGENT);
                    bad = 1; break;
                }
            }
            __builtin_amdgcn_s_sleep(2);
        }
#pragma unroll
        for (int o = 32; o >= 1; o >>= 1) gm = max(gm, (unsigned)__shfl_xor((int)gm, o));
        if (lane == 0) {
            *(volatile LAS unsigned*)(lds + CTL_GMAX + (wave - 4) * 4) = gm;
            if (bad) *(volatile LAS unsigned*)(lds + CTL_BAD) = 1u;
        }
    }
    if (dead && lane == 0) *(volatile LAS unsigned*)(lds + CTL_BAD) = 1u;
    asm volatile("s_waitcnt lgkmcnt(0)" ::: "memory");
    __builtin_amdgcn_s_barrier();

    if (wave < 4) {
        const int wm = wave & 1, wn = wave >> 1;
        const v4u g4 = *(const volatile LAS v4u*)(lds + CTL_GMAX);
        const unsigned bad = *(const volatile LAS unsigned*)(lds + CTL_BAD);
        const float amax = __builtin_bit_cast(float, max(max(g4.x, g4.y), max(g4.z, g4.w)));
        const float scale = amax / 448.0f;
        const float inv = amax > 0.f ? 1.0f / scale : 0.f;
        const __amdgpu_buffer_rsrc_t ro = __builtin_amdgcn_make_buffer_rsrc((void*)p.out[dest], 0, NR * MLOC * NLOC, 0x00020000);
        const unsigned voff = (unsigned)(p.rank * MLOC + mt * BM + wm * 128 + (lane & 15)) * NLOC + (unsigned)(ntl * BN + wn * 64 + (lane >> 4) * 16);
#pragma unroll
        for (int i = 0; i < 8; ++i) {
            v4u o;
#pragma unroll
            for (int rg = 0; rg < 4; ++rg) {
                const float q0 = __builtin_amdgcn_fmed3f(acc[i][0][rg] * inv, 0.f, 448.f);
                const float q1 = __builtin_amdgcn_fmed3f(acc[i][1][rg] * inv, 0.f, 448.f);
                const float q2 = __builtin_amdgcn_fmed3f(acc[i][2][rg] * inv, 0.f, 448.f);
                const float q3 = __builtin_amdgcn_fmed3f(acc[i][3][rg] * inv, 0.f, 448.f);
                int w = __builtin_amdgcn_cvt_pk_fp8_f32(q0, q1, 0, false);
                w = __builtin_amdgcn_cvt_pk_fp8_f32(q2, q3, w, true);
                o[rg] = bad ? 0x7f7f7f7fu : (unsigned)w;
            }
            __builtin_amdgcn_raw_buffer_store_b128(o, ro, voff + (unsigned)i * 16u * NLOC, 0, 17);
        }
        asm volatile("s_waitcnt vmcnt(0)" ::: "memory");
    }
    __builtin_amdgcn_s_barrier();
    if (tid == 0) {
        GAS unsigned* f = (GAS unsigned*)(p.ws[dest] + WS_DONE + ((size_t)p.rank * 32 + idx) * 4);
        __hip_atomic_store(f, call, __ATOMIC_RELAXED, __HIP_MEMORY_SCOPE_SYSTEM);
        if (b == 0) __hip_atomic_store((GAS unsigned*)(myws + WS_CALL), call, __ATOMIC_RELAXED, __HIP_MEMORY_SCOPE_AGENT);
    }
    if (dest == p.rank && idx == 0 && wave == 7) {
        const __amdgpu_buffer_rsrc_t rs = __builtin_amdgcn_make_buffer_rsrc((void*)(myws + WS_DONE), 0, NR * 32 * 4, 0x00020000);
        const u64 t0 = rt();
        for (unsigned sp = 0;; ++sp) {
            asm volatile("" ::: "memory");
            const v4u g = __builtin_amdgcn_raw_buffer_load_b128(rs, lane * 16u, 0, 17);
            const bool ok = g.x >= call && g.y >= call && g.z >= call && g.w >= call;
            if (__all(ok)) break;
            if ((sp & 63u) == 63u) {
                if (__builtin_amdgcn_readfirstlane(__hip_atomic_load(tmo, __ATOMIC_RELAXED, __HIP_MEMORY_SCOPE_AGENT)) == call || rt() - t0 > DEADLINE) {
                    if (lane == 0) {
                        __hip_atomic_store(tmo, call, __ATOMIC_RELAXED, __HIP_MEMORY_SCOPE_AGENT);
                        p.out[p.rank][(size_t)p.rank * MLOC * NLOC] = 0x7f;
                    }
                    break;
                }
            }
            __builtin_amdgcn_s_sleep(2);
        }
        __builtin_amdgcn_fence(__ATOMIC_ACQUIRE, "");
    }
}

extern "C" void kernel_launch(int rank, int nranks, ncclComm_t comm,
                              hipStream_t stream,
                              void* const* d_in, const int* in_sizes, int n_in,
                              void* d_out, int out_size,
                              const int* in_dtypes, int out_dtype,
                              void* const* d_out_all,
                              void* const* d_ws_all, size_t ws_bytes) {
    if (nranks != NR || (unsigned)rank >= (unsigned)NR || n_in < 2 || in_sizes[0] != MLOC * KDIM || in_sizes[1] != KDIM * NDIM ||
        out_size != NR * MLOC * NLOC || ws_bytes < WS_END) {
        fprintf(stderr, "kernel_launch: unexpected shapes\n");
        return;
    }
    HIP_CHECK(hipFuncSetAttribute((const void*)fused_kernel, hipFuncAttributeMaxDynamicSharedMemorySize, LDS_BYTES));
    Params p;
    p.x = (const float*)d_in[0];
    p.w = (const float*)d_in[1];
    for (int i = 0; i < NR; ++i) { p.out[i] = (unsigned char*)d_out_all[i]; p.ws[i] = (unsigned char*)d_ws_all[i]; }
    p.rank = rank;
    p.phase = 0;
    hipLaunchKernelGGL(fused_kernel, dim3(NWG), dim3(NTHREADS), 0, stream, p);
    p.phase = 1;
    hipLaunchKernelGGL(fused_kernel, dim3(NWG), dim3(NTHREADS), LDS_BYTES, stream, p);
    HIP_CHECK(hipGetLastError());
}
```

```cpp
#include <hip/hip_runtime.h>
#include <rccl/rccl.h>
#include <cstdio>
#include <cstdint>

#define HIP_CHECK(expr) do { hipError_t _e = (expr); if (_e != hipSuccess) { \
    fprintf(stderr, "HIP error %s at %s:%d\n", hipGetErrorString(_e), __FILE__, __LINE__); \
    return; } } while (0)

#define GAS __attribute__((address_space(1)))
#define LAS __attribute__((address_space(3)))
typedef float f32x4 __attribute__((ext_vector_type(4)));
typedef float f32x16 __attribute__((ext_vector_type(16)));
typedef float f32x2 __attribute__((ext_vector_type(2)));
typedef unsigned v4u __attribute__((ext_vector_type(4)));
typedef unsigned v2u __attribute__((ext_vector_type(2)));
typedef __bf16 bf16x2 __attribute__((ext_vector_type(2)));
typedef unsigned long long u64;

constexpr int NR = 8;
constexpr int MLOC = 512, KDIM = 4096, NDIM = 16384, NLOC = 2048;
constexpr int BM = 256, BN = 128, KH = 32;
constexpr int NH = KDIM / KH;
constexpr int NWG = 256;
constexpr int NTHREADS = 640;
#define CFG_T0 1500
#define CFG_XBLK 1000
#define CFG_WSTEP 330
#define CFG_WE 3
#define CFG_PAH 10
#define CFG_XDIR 2
#define CFG_XT1 1000
#define CFG_PUBSL 32
#define CFG_PSL 3
#define CFG_WPW 8
#define CFG_OPOL 16
#define CFG_WFS 64
#define CFG_WG 6
#define CFG_XCAP "24"
#define CFG_MS 3
#define CFG_EARLY 1
#define CFG_NB 11
constexpr int NA = 4, A_SLOT = 16384;
constexpr int NB = CFG_NB, B_SLOT = 8192;
constexpr int LDS_A = 0, LDS_B = NA * A_SLOT, LDS_CTL = LDS_B + NB * B_SLOT;
constexpr int CTL_XFULL = LDS_CTL;
constexpr int CTL_WFULL = LDS_CTL + 256;
constexpr int CTL_FREE = LDS_CTL + 2048;
constexpr int CTL_MISC = LDS_CTL + 512;
constexpr int CTL_DONE = CTL_MISC, CTL_GRDY = CTL_MISC + 4, CTL_BAD = CTL_MISC + 8, CTL_XCH = CTL_MISC + 32, CTL_XRDY = CTL_MISC + 64,
              CTL_CALL = CTL_MISC + 128, CTL_TILE = CTL_MISC + 132;
constexpr int CTL_JUNK = LDS_CTL + 1280;
constexpr int LDS_BYTES = LDS_CTL + 4096;
constexpr size_t WS_CALL = 0, WS_TMO = 128;
constexpr size_t WS_CLM = 4096;
constexpr size_t WS_LMAX = 8192;
constexpr size_t WS_GMAX = 16384;
constexpr size_t WS_XF = 131072;
constexpr size_t WS_XP = WS_XF + 16384;
constexpr int MS = CFG_MS;
constexpr int XDIR = CFG_XDIR;
constexpr size_t WS_WF = 262144;
constexpr int WFS = CFG_WFS;
constexpr size_t WS_SEQ = 524288;
constexpr size_t WS_XB = 1u << 20;
constexpr size_t WS_WB = WS_XB + (size_t)2 * NH * A_SLOT;
constexpr size_t WS_END = WS_WB + (size_t)16 * NH * B_SLOT;
constexpr u64 DEADLINE = 50000000ull;


struct Params {
    const float* x;
    const float* w;
    unsigned char* myws;
    int rank;
    unsigned srcmap;
    unsigned char* out;
    unsigned char* ws[NR];
};

__device__ __forceinline__ unsigned char* ws_of(const Params& p, int r) {
    u64 q = (u64)p.ws[0];
#pragma unroll
    for (int i = 1; i < NR; ++i) { u64 t = (u64)p.ws[i]; asm volatile("" : "+s"(t)); q = r == i ? t : q; }
    return (unsigned char*)q;
}
__device__ __forceinline__ u64 rt() {
    u64 t; asm volatile("s_memrealtime %0\n\ts_waitcnt lgkmcnt(0)" : "=s"(t) : : "memory"); return t;
}
__device__ __forceinline__ int lane_id() { return (int)__builtin_amdgcn_mbcnt_hi(~0u, __builtin_amdgcn_mbcnt_lo(~0u, 0u)); }
__device__ __forceinline__ unsigned cvt_bf16x2(float lo, float hi) {
    f32x2 v = {lo, hi};
    return __builtin_bit_cast(unsigned, __builtin_convertvector(v, bf16x2));
}
__device__ __forceinline__ int wave_max(int m) {
    m = max(m, __builtin_amdgcn_update_dpp(0, m, 0x128, 0xf, 0xf, false));
    m = max(m, __builtin_amdgcn_update_dpp(0, m, 0x124, 0xf, 0xf, false));
    m = max(m, __builtin_amdgcn_update_dpp(0, m, 0x122, 0xf, 0xf, false));
    m = max(m, __builtin_amdgcn_update_dpp(0, m, 0x121, 0xf, 0xf, false));
    return max(max(__builtin_amdgcn_readlane(m, 0), __builtin_amdgcn_readlane(m, 16)), max(__builtin_amdgcn_readlane(m, 32), __builtin_amdgcn_readlane(m, 48)));
}
__device__ __forceinline__ void lds_st_u32(unsigned addr, unsigned v) {
    asm volatile("ds_write_b32 %0, %1" : : "v"(addr), "v"(v) : "memory");
}
__device__ __forceinline__ bool all_ge(int seen, int need) { return __builtin_amdgcn_ballot_w64(seen < need) == 0ull; }
template <int SLEEP>
__device__ __forceinline__ void lds_wait_ge(unsigned addr, int need, int& seen, int& dead) {
    if (all_ge(seen, need) || dead) return;
#pragma nounroll
    for (unsigned sp = 0;; ++sp) {
        asm volatile("ds_read_b32 %0, %1\n\ts_waitcnt lgkmcnt(0)" : "=&v"(seen) : "v"(addr) : "memory");
        if (all_ge(seen, need)) break;
        __builtin_amdgcn_s_sleep(SLEEP);
        if (sp > (1u << 24) / (SLEEP + 1)) { dead = 1; break; }
    }
}
template <int SLEEP>
__device__ __forceinline__ int lds_wait_set(unsigned addr, int& dead) {
    int v = 0;
    lds_wait_ge<SLEEP>(addr, 1, v, dead);
    return __builtin_amdgcn_readfirstlane(v);
}
__device__ __forceinline__ bool timed_out(GAS unsigned* tmo, unsigned call, u64 t0) {
    if ((unsigned)__builtin_amdgcn_readfirstlane(__hip_atomic_load(tmo, __ATOMIC_RELAXED, __HIP_MEMORY_SCOPE_AGENT)) == call || rt() - t0 > DEADLINE) {
        __hip_atomic_store(tmo, call, __ATOMIC_RELAXED, __HIP_MEMORY_SCOPE_AGENT);
        return true;
    }
    return false;
}
#define BARRIER0() do { __builtin_amdgcn_sched_barrier(0); asm volatile("s_waitcnt lgkmcnt(0)" ::: "memory"); \
        __builtin_amdgcn_s_barrier(); __builtin_amdgcn_sched_barrier(0); } while (0)
#define UNTIL(ns, SL) do { while ((int)(rt() - tstart) < (ns) / 10) __builtin_amdgcn_s_sleep(SL); } while (0)
#define LOAD_CALL() __hip_atomic_load((GAS unsigned*)(myws + WS_CALL), __ATOMIC_RELAXED, __HIP_MEMORY_SCOPE_AGENT)
#define TILE_VARS(t) const int grp = ((t) >> 5) & 7, mt = (t) & 1, ct = ((t) >> 1) & 15, src = (p.srcmap >> (grp * 4)) & 7

__global__ void __launch_bounds__(NTHREADS, 3) fused_kernel(Params p) {
    extern __shared__ __attribute__((aligned(16))) unsigned char lds_raw[];
    LAS unsigned char* lds = (LAS unsigned char*)lds_raw;
    const int wave = __builtin_amdgcn_readfirstlane(threadIdx.x >> 6);
    int lane = threadIdx.x & 63;
    const int b = blockIdx.x;
    asm volatile("" : : "s"(p.x), "s"(p.w), "s"(p.myws), "s"(p.rank), "s"(p.ws[7]));
    unsigned char* myws = p.myws;
    GAS unsigned* tmo = (GAS unsigned*)(myws + WS_TMO);
    const int rot = (b + (b >> 4)) & 15;

    if (__builtin_expect(wave >= 8, 1)) {
        int lane;
        asm volatile("v_mbcnt_lo_u32_b32 %0, -1, 0\n\tv_mbcnt_hi_u32_b32 %0, -1, %0" : "=v"(lane));
        const unsigned vq = lane * 16;
        const int cm = rot & 1, fr = b >> 4, blk = (wave & 1) * 8 + 7 - (rot >> 1);
        f32x4 v[16];
        if (__builtin_expect(blk >= XDIR, 0)) {
            const u64 tstart = rt();
            __builtin_amdgcn_s_barrier();
            UNTIL(CFG_XT1 + (blk - XDIR) * CFG_XBLK, 2);
        }
        __builtin_amdgcn_sched_barrier(0);
        {
            const GAS f32x4* s = (const GAS f32x4*)(p.x + (size_t)(cm * BM + fr * 16 + (lane & 15)) * KDIM + blk * 8 * KH + (lane >> 4) * 4);
#pragma unroll
            for (int i = 0; i < 8; ++i) { v[2 * i] = __builtin_nontemporal_load(s + i * 8); v[2 * i + 1] = __builtin_nontemporal_load(s + i * 8 + 4); }
        }
        const unsigned c0_ = LOAD_CALL();
        __builtin_amdgcn_sched_barrier(0);
        if (blk < XDIR) __builtin_amdgcn_s_barrier();
        __builtin_amdgcn_sched_barrier(0);
        GAS unsigned* fl = (GAS unsigned*)(ws_of(p, lane & 7) + WS_XF + (size_t)((p.rank * 2 + cm) * 256 + blk * 16 + fr) * 4);
        const __amdgpu_buffer_rsrc_t drs = __builtin_amdgcn_make_buffer_rsrc((void*)(myws + WS_XB + (size_t)(cm * NH + blk * 8) * A_SLOT + fr * 1024), 0, 7 * A_SLOT + 1024, 0x00020000);
#pragma unroll
        for (int i = 0; i < 8; ++i) {
            v4u u;
            u.x = cvt_bf16x2(v[2 * i].x, v[2 * i].y); u.y = cvt_bf16x2(v[2 * i].z, v[2 * i].w);
            u.z = cvt_bf16x2(v[2 * i + 1].x, v[2 * i + 1].y); u.w = cvt_bf16x2(v[2 * i + 1].z, v[2 * i + 1].w);
            __builtin_amdgcn_raw_buffer_store_b128(u, drs, vq, i * A_SLOT, 17);
        }
        asm volatile("s_waitcnt vmcnt(0)" ::: "memory");
        if (blk < XDIR ? lane < NR : lane == p.rank) __hip_atomic_store(fl, c0_ + 1u, __ATOMIC_RELAXED, __HIP_MEMORY_SCOPE_SYSTEM);
        return;
    }
    const u64 tstart = rt();
    int dead = 0;
    const unsigned vq = lane * 16;
    *(LAS v2u*)(lds + LDS_CTL + threadIdx.x * 8) = (v2u){0u, 0u};

    if (wave < 4) {
        f32x16 acc[8];
        if (MS && wave == 0 && (rot >> 1) == 4) {
            const int cm = rot & 1, fr = b >> 4;
            f32x4 v[2 * MS + 1];
            const GAS f32x4* s = (const GAS f32x4*)(p.x + (size_t)(cm * BM + fr * 16 + (lane & 15)) * KDIM + (lane >> 4) * 4);
#pragma unroll
            for (int i = 0; i < MS; ++i) { v[2 * i] = s[i * 8]; v[2 * i + 1] = s[i * 8 + 4]; }
            const unsigned c0_ = LOAD_CALL();
            BARRIER0();
            GAS unsigned* fl = (GAS unsigned*)(ws_of(p, lane & 7) + WS_XP + (size_t)(p.rank * 2 + cm) * 1024 + 128 + fr * 4);
            const __amdgpu_buffer_rsrc_t drs = __builtin_amdgcn_make_buffer_rsrc((void*)(myws + WS_XB + (size_t)(cm * NH) * A_SLOT + fr * 1024), 0, 7 * A_SLOT + 1024, 0x00020000);
#pragma unroll
            for (int i = 0; i < MS; ++i) {
                v4u u;
                u.x = cvt_bf16x2(v[2 * i].x, v[2 * i].y); u.y = cvt_bf16x2(v[2 * i].z, v[2 * i].w);
                u.z = cvt_bf16x2(v[2 * i + 1].x, v[2 * i + 1].y); u.w = cvt_bf16x2(v[2 * i + 1].z, v[2 * i + 1].w);
                __builtin_amdgcn_raw_buffer_store_b128(u, drs, vq, i * A_SLOT, 17);
            }
            asm volatile("s_waitcnt vmcnt(0)" ::: "memory");
            if (lane < NR) __hip_atomic_store(fl, c0_ + 1u, __ATOMIC_RELAXED, __HIP_MEMORY_SCOPE_SYSTEM);
        } else
        BARRIER0();
        const int wm = wave & 1, wn = wave >> 1;
#pragma nounroll
        for (int pass = wave == 1 || wave == 2 ? 0 : 1; pass < 2; ++pass) {
        if (!pass && wave == 2) {
            asm volatile("" : "={v[40:55]}"(acc[0]), "={v[56:71]}"(acc[1]), "={v[72:87]}"(acc[2]), "={v[88:103]}"(acc[3]),
                  "={v[104:119]}"(acc[4]), "={v[120:135]}"(acc[5]), "={v[136:151]}"(acc[6]), "={v[152:167]}"(acc[7]));
        } else {
        asm volatile("v_mbcnt_lo_u32_b32 %0, -1, 0\n\tv_mbcnt_hi_u32_b32 %0, -1, %0" : "=v"(lane));
        const unsigned va = LDS_A + wm * 8192 + lane * 16;
        const unsigned bbase = LDS_B + wn * 4096 + lane * 16;
        const unsigned vca = (pass ? ((lane & 1) ? CTL_WFULL : CTL_XFULL) : CTL_JUNK) + wave * 4;
        if (!pass) lds_st_u32(vca, 0x1000u);
        lds_wait_set<2>(vca, dead);
        {
            unsigned sb = 0, need = pass ? 1 : NH - 3, cnt, lim = dead ? 0u : (1u << 23), nst = 0;
            asm volatile(
    "v_mov_b32 v5, %[bbase]\n\t"
    "v_mov_b32 v7, 1\n\t"
    "v_mov_b32 v40, 0\n\t"
    "v_mov_b32 v41, 0\n\t"
    "v_mov_b32 v42, 0\n\t"
    "v_mov_b32 v43, 0\n\t"
    "v_mov_b32 v44, 0\n\t"
    "v_mov_b32 v45, 0\n\t"
    "v_mov_b32 v46, 0\n\t"
    "v_mov_b32 v47, 0\n\t"
    "v_mov_b32 v48, 0\n\t"
    "v_mov_b32 v49, 0\n\t"
    "v_mov_b32 v50, 0\n\t"
    "v_mov_b32 v51, 0\n\t"
    "v_mov_b32 v52, 0\n\t"
    "v_mov_b32 v53, 0\n\t"
    "v_mov_b32 v54, 0\n\t"
    "v_mov_b32 v55, 0\n\t"
    "v_mov_b32 v56, 0\n\t"
    "v_mov_b32 v57, 0\n\t"
    "v_mov_b32 v58, 0\n\t"
    "v_mov_b32 v59, 0\n\t"
    "v_mov_b32 v60, 0\n\t"
    "v_mov_b32 v61, 0\n\t"
    "v_mov_b32 v62, 0\n\t"
    "v_mov_b32 v63, 0\n\t"
    "v_mov_b32 v64, 0\n\t"
    "v_mov_b32 v65, 0\n\t"
    "v_mov_b32 v66, 0\n\t"
    "v_mov_b32 v67, 0\n\t"
    "v_mov_b32 v68, 0\n\t"
    "v_mov_b32 v69, 0\n\t"
    "v_mov_b32 v70, 0\n\t"
    "v_mov_b32 v71, 0\n\t"
    "v_mov_b32 v72, 0\n\t"
    "v_mov_b32 v73, 0\n\t"
    "v_mov_b32 v74, 0\n\t"
    "v_mov_b32 v75, 0\n\t"
    "v_mov_b32 v76, 0\n\t"
    "v_mov_b32 v77, 0\n\t"
    "v_mov_b32 v78, 0\n\t"
    "v_mov_b32 v79, 0\n\t"
    "v_mov_b32 v80, 0\n\t"
    "v_mov_b32 v81, 0\n\t"
    "v_mov_b32 v82, 0\n\t"
    "v_mov_b32 v83, 0\n\t"
    "v_mov_b32 v84, 0\n\t"
    "v_mov_b32 v85, 0\n\t"
    "v_mov_b32 v86, 0\n\t"
    "v_mov_b32 v87, 0\n\t"
    "v_mov_b32 v88, 0\n\t"
    "v_mov_b32 v89, 0\n\t"
    "v_mov_b32 v90, 0\n\t"
    "v_mov_b32 v91, 0\n\t"
    "v_mov_b32 v92, 0\n\t"
    "v_mov_b32 v93, 0\n\t"
    "v_mov_b32 v94, 0\n\t"
    "v_mov_b32 v95, 0\n\t"
    "v_mov_b32 v96, 0\n\t"
    "v_mov_b32 v97, 0\n\t"
    "v_mov_b32 v98, 0\n\t"
    "v_mov_b32 v99, 0\n\t"
    "v_mov_b32 v100, 0\n\t"
    "v_mov_b32 v101, 0\n\t"
    "v_mov_b32 v102, 0\n\t"
    "v_mov_b32 v103, 0\n\t"
    "v_mov_b32 v104, 0\n\t"
    "v_mov_b32 v105, 0\n\t"
    "v_mov_b32 v106, 0\n\t"
    "v_mov_b32 v107, 0\n\t"
    "v_mov_b32 v108, 0\n\t"
    "v_mov_b32 v109, 0\n\t"
    "v_mov_b32 v110, 0\n\t"
    "v_mov_b32 v111, 0\n\t"
    "v_mov_b32 v112, 0\n\t"
    "v_mov_b32 v113, 0\n\t"
    "v_mov_b32 v114, 0\n\t"
    "v_mov_b32 v115, 0\n\t"
    "v_mov_b32 v116, 0\n\t"
    "v_mov_b32 v117, 0\n\t"
    "v_mov_b32 v118, 0\n\t"
    "v_mov_b32 v119, 0\n\t"
    "v_mov_b32 v120, 0\n\t"
    "v_mov_b32 v121, 0\n\t"
    "v_mov_b32 v122, 0\n\t"
    "v_mov_b32 v123, 0\n\t"
    "v_mov_b32 v124, 0\n\t"
    "v_mov_b32 v125, 0\n\t"
    "v_mov_b32 v126, 0\n\t"
    "v_mov_b32 v127, 0\n\t"
    "v_mov_b32 v128, 0\n\t"
    "v_mov_b32 v129, 0\n\t"
    "v_mov_b32 v130, 0\n\t"
    "v_mov_b32 v131, 0\n\t"
    "v_mov_b32 v132, 0\n\t"
    "v_mov_b32 v133, 0\n\t"
    "v_mov_b32 v134, 0\n\t"
    "v_mov_b32 v135, 0\n\t"
    "v_mov_b32 v136, 0\n\t"
    "v_mov_b32 v137, 0\n\t"
    "v_mov_b32 v138, 0\n\t"
    "v_mov_b32 v139, 0\n\t"
    "v_mov_b32 v140, 0\n\t"
    "v_mov_b32 v141, 0\n\t"
    "v_mov_b32 v142, 0\n\t"
    "v_mov_b32 v143, 0\n\t"
    "v_mov_b32 v144, 0\n\t"
    "v_mov_b32 v145, 0\n\t"
    "v_mov_b32 v146, 0\n\t"
    "v_mov_b32 v147, 0\n\t"
    "v_mov_b32 v148, 0\n\t"
    "v_mov_b32 v149, 0\n\t"
    "v_mov_b32 v150, 0\n\t"
    "v_mov_b32 v151, 0\n\t"
    "v_mov_b32 v152, 0\n\t"
    "v_mov_b32 v153, 0\n\t"
    "v_mov_b32 v154, 0\n\t"
    "v_mov_b32 v155, 0\n\t"
    "v_mov_b32 v156, 0\n\t"
    "v_mov_b32 v157, 0\n\t"
    "v_mov_b32 v158, 0\n\t"
    "v_mov_b32 v159, 0\n\t"
    "v_mov_b32 v160, 0\n\t"
    "v_mov_b32 v161, 0\n\t"
    "v_mov_b32 v162, 0\n\t"
    "v_mov_b32 v163, 0\n\t"
    "v_mov_b32 v164, 0\n\t"
    "v_mov_b32 v165, 0\n\t"
    "v_mov_b32 v166, 0\n\t"
    "v_mov_b32 v167, 0\n\t"
    "ds_read_b128 v[24:27], %[va] offset:0\n\t"
    "ds_read_b128 v[28:31], %[va] offset:1024\n\t"
    "ds_read_b128 v[8:11], v5 offset:0\n\t"
    "ds_read_b128 v[12:15], v5 offset:1024\n\t"
    "ds_read_b128 v[16:19], v5 offset:2048\n\t"
    "ds_read_b128 v[20:23], v5 offset:3072\n\t"
    "ds_write_b32 %[vca], v7 offset:2048\n\t"
    "LLOOP_%=:\n\t"
    "s_waitcnt lgkmcnt(4)\n\t"
    "v_mfma_f32_16x16x32_bf16 v[40:43], v[8:11], v[24:27], v[40:43]\n\t"
    "ds_read_b128 v[32:35], %[va] offset:2048\n\t"
    "v_mfma_f32_16x16x32_bf16 v[56:59], v[8:11], v[28:31], v[56:59]\n\t"
    "ds_read_b128 v[36:39], %[va] offset:3072\n\t"
    "s_waitcnt lgkmcnt(5)\n\t"
    "v_mfma_f32_16x16x32_bf16 v[44:47], v[12:15], v[24:27], v[44:47]\n\t"
    "v_add_u32 v7, 1, v7\n\t"
    "v_mfma_f32_16x16x32_bf16 v[60:63], v[12:15], v[28:31], v[60:63]\n\t"
    "s_add_u32 %[sb], %[sb], %[bslot]\n\t"
    "s_waitcnt lgkmcnt(4)\n\t"
    "v_mfma_f32_16x16x32_bf16 v[48:51], v[16:19], v[24:27], v[48:51]\n\t"
    "s_cmp_lg_u32 %[sb], %[bend]\n\t"
    "v_mfma_f32_16x16x32_bf16 v[64:67], v[16:19], v[28:31], v[64:67]\n\t"
    "s_cselect_b32 %[sb], %[sb], 0\n\t"
    "s_waitcnt lgkmcnt(3)\n\t"
    "v_mfma_f32_16x16x32_bf16 v[52:55], v[20:23], v[24:27], v[52:55]\n\t"
    "v_add_u32 v5, %[sb], %[bbase]\n\t"
    "v_mfma_f32_16x16x32_bf16 v[68:71], v[20:23], v[28:31], v[68:71]\n\t"
    "s_waitcnt lgkmcnt(0)\n\t"
    "v_mfma_f32_16x16x32_bf16 v[72:75], v[8:11], v[32:35], v[72:75]\n\t"
    "ds_read_b128 v[24:27], %[va] offset:4096\n\t"
    "v_mfma_f32_16x16x32_bf16 v[88:91], v[8:11], v[36:39], v[88:91]\n\t"
    "ds_read_b128 v[28:31], %[va] offset:5120\n\t"
    "v_mfma_f32_16x16x32_bf16 v[76:79], v[12:15], v[32:35], v[76:79]\n\t"
    "s_add_u32 %[need], %[need], 1\n\t"
    "v_mfma_f32_16x16x32_bf16 v[92:95], v[12:15], v[36:39], v[92:95]\n\t"
    "v_mfma_f32_16x16x32_bf16 v[80:83], v[16:19], v[32:35], v[80:83]\n\t"
    "v_mfma_f32_16x16x32_bf16 v[96:99], v[16:19], v[36:39], v[96:99]\n\t"
    "v_mfma_f32_16x16x32_bf16 v[84:87], v[20:23], v[32:35], v[84:87]\n\t"
    "v_mfma_f32_16x16x32_bf16 v[100:103], v[20:23], v[36:39], v[100:103]\n\t"
    "s_waitcnt lgkmcnt(0)\n\t"
    "v_mfma_f32_16x16x32_bf16 v[104:107], v[8:11], v[24:27], v[104:107]\n\t"
    "ds_read_b128 v[32:35], %[va] offset:6144\n\t"
    "v_mfma_f32_16x16x32_bf16 v[120:123], v[8:11], v[28:31], v[120:123]\n\t"
    "ds_read_b128 v[36:39], %[va] offset:7168\n\t"
    "v_mfma_f32_16x16x32_bf16 v[108:111], v[12:15], v[24:27], v[108:111]\n\t"
    "ds_write_b32 %[vca], v7 offset:2048\n\t"
    "v_mfma_f32_16x16x32_bf16 v[124:127], v[12:15], v[28:31], v[124:127]\n\t"
    "ds_read_b32 v6, %[vca] offset:0\n\t"
    "v_mfma_f32_16x16x32_bf16 v[112:115], v[16:19], v[24:27], v[112:115]\n\t"
    "v_add_u32 v7, 1, v7\n\t"
    "v_mfma_f32_16x16x32_bf16 v[128:131], v[16:19], v[28:31], v[128:131]\n\t"
    "v_mfma_f32_16x16x32_bf16 v[116:119], v[20:23], v[24:27], v[116:119]\n\t"
    "v_mfma_f32_16x16x32_bf16 v[132:135], v[20:23], v[28:31], v[132:135]\n\t"
    "s_waitcnt lgkmcnt(0)\n\t"
    "v_cmp_gt_u32 vcc, %[need], v6\n\t"
    "v_mfma_f32_16x16x32_bf16 v[136:139], v[8:11], v[32:35], v[136:139]\n\t"
    "v_mfma_f32_16x16x32_bf16 v[152:155], v[8:11], v[36:39], v[152:155]\n\t"
    "s_cbranch_vccnz LSLOW0_%=\n\t"
    "LBACK0_%=:\n\t"
    "ds_read_b128 v[24:27], %[va] offset:16384\n\t"
    "ds_read_b128 v[28:31], %[va] offset:17408\n\t"
    "ds_read_b128 v[8:11], v5 offset:0\n\t"
    "v_mfma_f32_16x16x32_bf16 v[140:143], v[12:15], v[32:35], v[140:143]\n\t"
    "v_mfma_f32_16x16x32_bf16 v[156:159], v[12:15], v[36:39], v[156:159]\n\t"
    "ds_read_b128 v[12:15], v5 offset:1024\n\t"
    "v_mfma_f32_16x16x32_bf16 v[144:147], v[16:19], v[32:35], v[144:147]\n\t"
    "v_mfma_f32_16x16x32_bf16 v[160:163], v[16:19], v[36:39], v[160:163]\n\t"
    "ds_read_b128 v[16:19], v5 offset:2048\n\t"
    "v_mfma_f32_16x16x32_bf16 v[148:151], v[20:23], v[32:35], v[148:151]\n\t"
    "v_mfma_f32_16x16x32_bf16 v[164:167], v[20:23], v[36:39], v[164:167]\n\t"
    "ds_read_b128 v[20:23], v5 offset:3072\n\t"
    "ds_write_b32 %[vca], v7 offset:2048\n\t"
    "s_waitcnt lgkmcnt(4)\n\t"
    "v_mfma_f32_16x16x32_bf16 v[40:43], v[8:11], v[24:27], v[40:43]\n\t"
    "ds_read_b128 v[32:35], %[va] offset:18432\n\t"
    "v_mfma_f32_16x16x32_bf16 v[56:59], v[8:11], v[28:31], v[56:59]\n\t"
    "ds_read_b128 v[36:39], %[va] offset:19456\n\t"
    "s_waitcnt lgkmcnt(5)\n\t"
    "v_mfma_f32_16x16x32_bf16 v[44:47], v[12:15], v[24:27], v[44:47]\n\t"
    "v_add_u32 v7, 1, v7\n\t"
    "v_mfma_f32_16x16x32_bf16 v[60:63], v[12:15], v[28:31], v[60:63]\n\t"
    "s_add_u32 %[sb], %[sb], %[bslot]\n\t"
    "s_waitcnt lgkmcnt(4)\n\t"
    "v_mfma_f32_16x16x32_bf16 v[48:51], v[16:19], v[24:27], v[48:51]\n\t"
    "s_cmp_lg_u32 %[sb], %[bend]\n\t"
    "v_mfma_f32_16x16x32_bf16 v[64:67], v[16:19], v[28:31], v[64:67]\n\t"
    "s_cselect_b32 %[sb], %[sb], 0\n\t"
    "s_waitcnt lgkmcnt(3)\n\t"
    "v_mfma_f32_16x16x32_bf16 v[52:55], v[20:23], v[24:27], v[52:55]\n\t"
    "v_add_u32 v5, %[sb], %[bbase]\n\t"
    "v_mfma_f32_16x16x32_bf16 v[68:71], v[20:23], v[28:31], v[68:71]\n\t"
    "s_waitcnt lgkmcnt(0)\n\t"
    "v_mfma_f32_16x16x32_bf16 v[72:75], v[8:11], v[32:35], v[72:75]\n\t"
    "ds_read_b128 v[24:27], %[va] offset:20480\n\t"
    "v_mfma_f32_16x16x32_bf16 v[88:91], v[8:11], v[36:39], v[88:91]\n\t"
    "ds_read_b128 v[28:31], %[va] offset:21504\n\t"
    "v_mfma_f32_16x16x32_bf16 v[76:79], v[12:15], v[32:35], v[76:79]\n\t"
    "s_add_u32 %[need], %[need], 1\n\t"
    "v_mfma_f32_16x16x32_bf16 v[92:95], v[12:15], v[36:39], v[92:95]\n\t"
    "v_mfma_f32_16x16x32_bf16 v[80:83], v[16:19], v[32:35], v[80:83]\n\t"
    "v_mfma_f32_16x16x32_bf16 v[96:99], v[16:19], v[36:39], v[96:99]\n\t"
    "v_mfma_f32_16x16x32_bf16 v[84:87], v[20:23], v[32:35], v[84:87]\n\t"
    "v_mfma_f32_16x16x32_bf16 v[100:103], v[20:23], v[36:39], v[100:103]\n\t"
    "s_waitcnt lgkmcnt(0)\n\t"
    "v_mfma_f32_16x16x32_bf16 v[104:107], v[8:11], v[24:27], v[104:107]\n\t"
    "ds_read_b128 v[32:35], %[va] offset:22528\n\t"
    "v_mfma_f32_16x16x32_bf16 v[120:123], v[8:11], v[28:31], v[120:123]\n\t"
    "ds_read_b128 v[36:39], %[va] offset:23552\n\t"
    "v_mfma_f32_16x16x32_bf16 v[108:111], v[12:15], v[24:27], v[108:111]\n\t"
    "ds_write_b32 %[vca], v7 offset:2048\n\t"
    "v_mfma_f32_16x16x32_bf16 v[124:127], v[12:15], v[28:31], v[124:127]\n\t"
    "ds_read_b32 v6, %[vca] offset:0\n\t"
    "v_mfma_f32_16x16x32_bf16 v[112:115], v[16:19], v[24:27], v[112:115]\n\t"
    "v_add_u32 v7, 1, v7\n\t"
    "v_mfma_f32_16x16x32_bf16 v[128:131], v[16:19], v[28:31], v[128:131]\n\t"
    "v_mfma_f32_16x16x32_bf16 v[116:119], v[20:23], v[24:27], v[116:119]\n\t"
    "v_mfma_f32_16x16x32_bf16 v[132:135], v[20:23], v[28:31], v[132:135]\n\t"
    "s_waitcnt lgkmcnt(0)\n\t"
    "v_cmp_gt_u32 vcc, %[need], v6\n\t"
    "v_mfma_f32_16x16x32_bf16 v[136:139], v[8:11], v[32:35], v[136:139]\n\t"
    "v_mfma_f32_16x16x32_bf16 v[152:155], v[8:11], v[36:39], v[152:155]\n\t"
    "s_cbranch_vccnz LSLOW1_%=\n\t"
    "LBACK1_%=:\n\t"
    "ds_read_b128 v[24:27], %[va] offset:32768\n\t"
    "ds_read_b128 v[28:31], %[va] offset:33792\n\t"
    "ds_read_b128 v[8:11], v5 offset:0\n\t"
    "v_mfma_f32_16x16x32_bf16 v[140:143], v[12:15], v[32:35], v[140:143]\n\t"
    "v_mfma_f32_16x16x32_bf16 v[156:159], v[12:15], v[36:39], v[156:159]\n\t"
    "ds_read_b128 v[12:15], v5 offset:1024\n\t"
    "v_mfma_f32_16x16x32_bf16 v[144:147], v[16:19], v[32:35], v[144:147]\n\t"
    "v_mfma_f32_16x16x32_bf16 v[160:163], v[16:19], v[36:39], v[160:163]\n\t"
    "ds_read_b128 v[16:19], v5 offset:2048\n\t"
    "v_mfma_f32_16x16x32_bf16 v[148:151], v[20:23], v[32:35], v[148:151]\n\t"
    "v_mfma_f32_16x16x32_bf16 v[164:167], v[20:23], v[36:39], v[164:167]\n\t"
    "ds_read_b128 v[20:23], v5 offset:3072\n\t"
    "ds_write_b32 %[vca], v7 offset:2048\n\t"
    "s_waitcnt lgkmcnt(4)\n\t"
    "v_mfma_f32_16x16x32_bf16 v[40:43], v[8:11], v[24:27], v[40:43]\n\t"
    "ds_read_b128 v[32:35], %[va] offset:34816\n\t"
    "v_mfma_f32_16x16x32_bf16 v[56:59], v[8:11], v[28:31], v[56:59]\n\t"
    "ds_read_b128 v[36:39], %[va] offset:35840\n\t"
    "s_waitcnt lgkmcnt(5)\n\t"
    "v_mfma_f32_16x16x32_bf16 v[44:47], v[12:15], v[24:27], v[44:47]\n\t"
    "v_add_u32 v7, 1, v7\n\t"
    "v_mfma_f32_16x16x32_bf16 v[60:63], v[12:15], v[28:31], v[60:63]\n\t"
    "s_add_u32 %[sb], %[sb], %[bslot]\n\t"
    "s_waitcnt lgkmcnt(4)\n\t"
    "v_mfma_f32_16x16x32_bf16 v[48:51], v[16:19], v[24:27], v[48:51]\n\t"
    "s_cmp_lg_u32 %[sb], %[bend]\n\t"
    "v_mfma_f32_16x16x32_bf16 v[64:67], v[16:19], v[28:31], v[64:67]\n\t"
    "s_cselect_b32 %[sb], %[sb], 0\n\t"
    "s_waitcnt lgkmcnt(3)\n\t"
    "v_mfma_f32_16x16x32_bf16 v[52:55], v[20:23], v[24:27], v[52:55]\n\t"
    "v_add_u32 v5, %[sb], %[bbase]\n\t"
    "v_mfma_f32_16x16x32_bf16 v[68:71], v[20:23], v[28:31], v[68:71]\n\t"
    "s_waitcnt lgkmcnt(0)\n\t"
    "v_mfma_f32_16x16x32_bf16 v[72:75], v[8:11], v[32:35], v[72:75]\n\t"
    "ds_read_b128 v[24:27], %[va] offset:36864\n\t"
    "v_mfma_f32_16x16x32_bf16 v[88:91], v[8:11], v[36:39], v[88:91]\n\t"
    "ds_read_b128 v[28:31], %[va] offset:37888\n\t"
    "v_mfma_f32_16x16x32_bf16 v[76:79], v[12:15], v[32:35], v[76:79]\n\t"
    "s_add_u32 %[need], %[need], 1\n\t"
    "v_mfma_f32_16x16x32_bf16 v[92:95], v[12:15], v[36:39], v[92:95]\n\t"
    "v_mfma_f32_16x16x32_bf16 v[80:83], v[16:19], v[32:35], v[80:83]\n\t"
    "v_mfma_f32_16x16x32_bf16 v[96:99], v[16:19], v[36:39], v[96:99]\n\t"
    "v_mfma_f32_16x16x32_bf16 v[84:87], v[20:23], v[32:35], v[84:87]\n\t"
    "v_mfma_f32_16x16x32_bf16 v[100:103], v[20:23], v[36:39], v[100:103]\n\t"
    "s_waitcnt lgkmcnt(0)\n\t"
    "v_mfma_f32_16x16x32_bf16 v[104:107], v[8:11], v[24:27], v[104:107]\n\t"
    "ds_read_b128 v[32:35], %[va] offset:38912\n\t"
    "v_mfma_f32_16x16x32_bf16 v[120:123], v[8:11], v[28:31], v[120:123]\n\t"
    "ds_read_b128 v[36:39], %[va] offset:39936\n\t"
    "v_mfma_f32_16x16x32_bf16 v[108:111], v[12:15], v[24:27], v[108:111]\n\t"
    "ds_write_b32 %[vca], v7 offset:2048\n\t"
    "v_mfma_f32_16x16x32_bf16 v[124:127], v[12:15], v[28:31], v[124:127]\n\t"
    "ds_read_b32 v6, %[vca] offset:0\n\t"
    "v_mfma_f32_16x16x32_bf16 v[112:115], v[16:19], v[24:27], v[112:115]\n\t"
    "v_add_u32 v7, 1, v7\n\t"
    "v_mfma_f32_16x16x32_bf16 v[128:131], v[16:19], v[28:31], v[128:131]\n\t"
    "v_mfma_f32_16x16x32_bf16 v[116:119], v[20:23], v[24:27], v[116:119]\n\t"
    "v_mfma_f32_16x16x32_bf16 v[132:135], v[20:23], v[28:31], v[132:135]\n\t"
    "s_waitcnt lgkmcnt(0)\n\t"
    "v_cmp_gt_u32 vcc, %[need], v6\n\t"
    "v_mfma_f32_16x16x32_bf16 v[136:139], v[8:11], v[32:35], v[136:139]\n\t"
    "v_mfma_f32_16x16x32_bf16 v[152:155], v[8:11], v[36:39], v[152:155]\n\t"
    "s_cbranch_vccnz LSLOW2_%=\n\t"
    "LBACK2_%=:\n\t"
    "ds_read_b128 v[24:27], %[va] offset:49152\n\t"
    "ds_read_b128 v[28:31], %[va] offset:50176\n\t"
    "ds_read_b128 v[8:11], v5 offset:0\n\t"
    "v_mfma_f32_16x16x32_bf16 v[140:143], v[12:15], v[32:35], v[140:143]\n\t"
    "v_mfma_f32_16x16x32_bf16 v[156:159], v[12:15], v[36:39], v[156:159]\n\t"
    "ds_read_b128 v[12:15], v5 offset:1024\n\t"
    "v_mfma_f32_16x16x32_bf16 v[144:147], v[16:19], v[32:35], v[144:147]\n\t"
    "v_mfma_f32_16x16x32_bf16 v[160:163], v[16:19], v[36:39], v[160:163]\n\t"
    "ds_read_b128 v[16:19], v5 offset:2048\n\t"
    "v_mfma_f32_16x16x32_bf16 v[148:151], v[20:23], v[32:35], v[148:151]\n\t"
    "v_mfma_f32_16x16x32_bf16 v[164:167], v[20:23], v[36:39], v[164:167]\n\t"
    "ds_read_b128 v[20:23], v5 offset:3072\n\t"
    "ds_write_b32 %[vca], v7 offset:2048\n\t"
    "s_waitcnt lgkmcnt(4)\n\t"
    "v_mfma_f32_16x16x32_bf16 v[40:43], v[8:11], v[24:27], v[40:43]\n\t"
    "ds_read_b128 v[32:35], %[va] offset:51200\n\t"
    "v_mfma_f32_16x16x32_bf16 v[56:59], v[8:11], v[28:31], v[56:59]\n\t"
    "ds_read_b128 v[36:39], %[va] offset:52224\n\t"
    "s_waitcnt lgkmcnt(5)\n\t"
    "v_mfma_f32_16x16x32_bf16 v[44:47], v[12:15], v[24:27], v[44:47]\n\t"
    "v_add_u32 v7, 1, v7\n\t"
    "v_mfma_f32_16x16x32_bf16 v[60:63], v[12:15], v[28:31], v[60:63]\n\t"
    "s_add_u32 %[sb], %[sb], %[bslot]\n\t"
    "s_waitcnt lgkmcnt(4)\n\t"
    "v_mfma_f32_16x16x32_bf16 v[48:51], v[16:19], v[24:27], v[48:51]\n\t"
    "s_cmp_lg_u32 %[sb], %[bend]\n\t"
    "v_mfma_f32_16x16x32_bf16 v[64:67], v[16:19], v[28:31], v[64:67]\n\t"
    "s_cselect_b32 %[sb], %[sb], 0\n\t"
    "s_waitcnt lgkmcnt(3)\n\t"
    "v_mfma_f32_16x16x32_bf16 v[52:55], v[20:23], v[24:27], v[52:55]\n\t"
    "v_add_u32 v5, %[sb], %[bbase]\n\t"
    "v_mfma_f32_16x16x32_bf16 v[68:71], v[20:23], v[28:31], v[68:71]\n\t"
    "s_waitcnt lgkmcnt(0)\n\t"
    "v_mfma_f32_16x16x32_bf16 v[72:75], v[8:11], v[32:35], v[72:75]\n\t"
    "ds_read_b128 v[24:27], %[va] offset:53248\n\t"
    "v_mfma_f32_16x16x32_bf16 v[88:91], v[8:11], v[36:39], v[88:91]\n\t"
    "ds_read_b128 v[28:31], %[va] offset:54272\n\t"
    "v_mfma_f32_16x16x32_bf16 v[76:79], v[12:15], v[32:35], v[76:79]\n\t"
    "s_add_u32 %[need], %[need], 1\n\t"
    "v_mfma_f32_16x16x32_bf16 v[92:95], v[12:15], v[36:39], v[92:95]\n\t"
    "v_mfma_f32_16x16x32_bf16 v[80:83], v[16:19], v[32:35], v[80:83]\n\t"
    "v_mfma_f32_16x16x32_bf16 v[96:99], v[16:19], v[36:39], v[96:99]\n\t"
    "v_mfma_f32_16x16x32_bf16 v[84:87], v[20:23], v[32:35], v[84:87]\n\t"
    "v_mfma_f32_16x16x32_bf16 v[100:103], v[20:23], v[36:39], v[100:103]\n\t"
    "s_waitcnt lgkmcnt(0)\n\t"
    "v_mfma_f32_16x16x32_bf16 v[104:107], v[8:11], v[24:27], v[104:107]\n\t"
    "ds_read_b128 v[32:35], %[va] offset:55296\n\t"
    "v_mfma_f32_16x16x32_bf16 v[120:123], v[8:11], v[28:31], v[120:123]\n\t"
    "ds_read_b128 v[36:39], %[va] offset:56320\n\t"
    "v_mfma_f32_16x16x32_bf16 v[108:111], v[12:15], v[24:27], v[108:111]\n\t"
    "ds_write_b32 %[vca], v7 offset:2048\n\t"
    "v_mfma_f32_16x16x32_bf16 v[124:127], v[12:15], v[28:31], v[124:127]\n\t"
    "ds_read_b32 v6, %[vca] offset:0\n\t"
    "v_mfma_f32_16x16x32_bf16 v[112:115], v[16:19], v[24:27], v[112:115]\n\t"
    "v_add_u32 v7, 1, v7\n\t"
    "v_mfma_f32_16x16x32_bf16 v[128:131], v[16:19], v[28:31], v[128:131]\n\t"
    "v_mfma_f32_16x16x32_bf16 v[116:119], v[20:23], v[24:27], v[116:119]\n\t"
    "v_mfma_f32_16x16x32_bf16 v[132:135], v[20:23], v[28:31], v[132:135]\n\t"
    "s_waitcnt lgkmcnt(0)\n\t"
    "v_cmp_gt_u32 vcc, %[need], v6\n\t"
    "v_mfma_f32_16x16x32_bf16 v[136:139], v[8:11], v[32:35], v[136:139]\n\t"
    "v_mfma_f32_16x16x32_bf16 v[152:155], v[8:11], v[36:39], v[152:155]\n\t"
    "s_cbranch_vccnz LSLOW3_%=\n\t"
    "LBACK3_%=:\n\t"
    "ds_read_b128 v[24:27], %[va] offset:0\n\t"
    "ds_read_b128 v[28:31], %[va] offset:1024\n\t"
    "ds_read_b128 v[8:11], v5 offset:0\n\t"
    "v_mfma_f32_16x16x32_bf16 v[140:143], v[12:15], v[32:35], v[140:143]\n\t"
    "v_mfma_f32_16x16x32_bf16 v[156:159], v[12:15], v[36:39], v[156:159]\n\t"
    "ds_read_b128 v[12:15], v5 offset:1024\n\t"
    "v_mfma_f32_16x16x32_bf16 v[144:147], v[16:19], v[32:35], v[144:147]\n\t"
    "v_mfma_f32_16x16x32_bf16 v[160:163], v[16:19], v[36:39], v[160:163]\n\t"
    "ds_read_b128 v[16:19], v5 offset:2048\n\t"
    "v_mfma_f32_16x16x32_bf16 v[148:151], v[20:23], v[32:35], v[148:151]\n\t"
    "v_mfma_f32_16x16x32_bf16 v[164:167], v[20:23], v[36:39], v[164:167]\n\t"
    "ds_read_b128 v[20:23], v5 offset:3072\n\t"
    "ds_write_b32 %[vca], v7 offset:2048\n\t"
    "s_cmp_lt_u32 %[need], %[last]\n\t"
    "s_cbranch_scc1 LLOOP_%=\n\t"
    "s_waitcnt lgkmcnt(0)\n\t"
    "s_nop 7\n\t"
    "s_nop 3\n\t"
    "s_branch LEND_%=\n\t"
    "LSLOW0_%=:\n\t"
    "s_add_u32 %[nst], %[nst], 1\n\t"
    "s_mov_b32 %[cnt], 0\n\t"
    "LSPIN0_%=:\n\t"
    "ds_read_b32 v6, %[vca] offset:0\n\t"
    "s_add_u32 %[cnt], %[cnt], 1\n\t"
    "s_waitcnt lgkmcnt(0)\n\t"
    "v_cmp_gt_u32 vcc, %[need], v6\n\t"
    "s_nop 4\n\t"
    "s_cbranch_vccz LBACK0_%=\n\t"
    "s_sleep 1\n\t"
    "s_cmp_lt_u32 %[cnt], %[lim]\n\t"
    "s_cbranch_scc1 LSPIN0_%=\n\t"
    "s_mov_b32 %[lim], 0\n\t"
    "s_branch LBACK0_%=\n\t"
    "LSLOW1_%=:\n\t"
    "s_add_u32 %[nst], %[nst], 1\n\t"
    "s_mov_b32 %[cnt], 0\n\t"
    "LSPIN1_%=:\n\t"
    "ds_read_b32 v6, %[vca] offset:0\n\t"
    "s_add_u32 %[cnt], %[cnt], 1\n\t"
    "s_waitcnt lgkmcnt(0)\n\t"
    "v_cmp_gt_u32 vcc, %[need], v6\n\t"
    "s_nop 4\n\t"
    "s_cbranch_vccz LBACK1_%=\n\t"
    "s_sleep 1\n\t"
    "s_cmp_lt_u32 %[cnt], %[lim]\n\t"
    "s_cbranch_scc1 LSPIN1_%=\n\t"
    "s_mov_b32 %[lim], 0\n\t"
    "s_branch LBACK1_%=\n\t"
    "LSLOW2_%=:\n\t"
    "s_add_u32 %[nst], %[nst], 1\n\t"
    "s_mov_b32 %[cnt], 0\n\t"
    "LSPIN2_%=:\n\t"
    "ds_read_b32 v6, %[vca] offset:0\n\t"
    "s_add_u32 %[cnt], %[cnt], 1\n\t"
    "s_waitcnt lgkmcnt(0)\n\t"
    "v_cmp_gt_u32 vcc, %[need], v6\n\t"
    "s_nop 4\n\t"
    "s_cbranch_vccz LBACK2_%=\n\t"
    "s_sleep 1\n\t"
    "s_cmp_lt_u32 %[cnt], %[lim]\n\t"
    "s_cbranch_scc1 LSPIN2_%=\n\t"
    "s_mov_b32 %[lim], 0\n\t"
    "s_branch LBACK2_%=\n\t"
    "LSLOW3_%=:\n\t"
    "s_add_u32 %[nst], %[nst], 1\n\t"
    "s_mov_b32 %[cnt], 0\n\t"
    "LSPIN3_%=:\n\t"
    "ds_read_b32 v6, %[vca] offset:0\n\t"
    "s_add_u32 %[cnt], %[cnt], 1\n\t"
    "s_waitcnt lgkmcnt(0)\n\t"
    "v_cmp_gt_u32 vcc, %[need], v6\n\t"
    "s_nop 4\n\t"
    "s_cbranch_vccz LBACK3_%=\n\t"
    "s_sleep 1\n\t"
    "s_cmp_lt_u32 %[cnt], %[lim]\n\t"
    "s_cbranch_scc1 LSPIN3_%=\n\t"
    "s_mov_b32 %[lim], 0\n\t"
    "s_branch LBACK3_%=\n\t"
    "LEND_%=:\n\t"
                : "=&{v[40:55]}"(acc[0]), "=&{v[56:71]}"(acc[1]), "=&{v[72:87]}"(acc[2]), "=&{v[88:103]}"(acc[3]),
                  "=&{v[104:119]}"(acc[4]), "=&{v[120:135]}"(acc[5]), "=&{v[136:151]}"(acc[6]), "=&{v[152:167]}"(acc[7]),
                  [sb] "+s"(sb), [need] "+s"(need), [cnt] "=&s"(cnt), [lim] "+s"(lim), [nst] "+s"(nst)
                : [va] "v"(va), [bbase] "v"(bbase), [vca] "v"(vca),
                  [bslot] "i"(B_SLOT), [bend] "i"(NB * B_SLOT), [last] "i"(NH + 1)
                : "memory", "scc", "vcc",
                  "v5", "v6", "v7", "v8", "v9", "v10", "v11", "v12", "v13", "v14", "v15", "v16", "v17", "v18", "v19", "v20",
                  "v21", "v22", "v23", "v24", "v25", "v26", "v27", "v28", "v29", "v30", "v31", "v32", "v33", "v34", "v35", "v36",
                  "v37", "v38", "v39"
            );
            if (lim == 0) dead = 1;
        }
        if (!pass) continue;
        }
        unsigned cm_;
        asm volatile("v_mbcnt_lo_u32_b32 %0, -1, 0\n\tv_mbcnt_hi_u32_b32 %0, -1, %0\n\tv_mov_b32 %1, %2" : "=v"(lane), "=v"(cm_) : "i"(CTL_MISC));
        const unsigned badw = *(const volatile LAS unsigned*)(lds + cm_ + (CTL_BAD - CTL_MISC));
        const v2u ct2 = *(const volatile LAS v2u*)(lds + cm_ + (CTL_CALL - CTL_MISC));
        int m = 0;
#pragma unroll
        for (int i = 0; i < 8; ++i)
#pragma unroll
            for (int c = 0; c < 16; c += 2) m = max(max(m, __float_as_int(acc[i][c])), __float_as_int(acc[i][c + 1]));
        if (badw | (unsigned)dead) m = (int)(cm_ + (0x7fc00000u - CTL_MISC));
        m = wave_max(m);
        __builtin_amdgcn_raw_buffer_store_b64((v2u){(unsigned)m, ct2.x}, __builtin_amdgcn_make_buffer_rsrc((void*)(myws + WS_LMAX + (size_t)(b * 4 + wave) * 8), 0, pass * 8, 0x00020000), lane * 8, 0, 16);
        lds_st_u32(cm_ + (CTL_DONE - CTL_MISC), (unsigned)pass);
#pragma unroll
        for (int i = 0; i < 8; ++i)
#pragma unroll
            for (int c = 0; c < 16; ++c) acc[i][c] = __int_as_float(max(__float_as_int(acc[i][c]), 0));
        TILE_VARS(__builtin_amdgcn_readfirstlane((int)ct2.y) - 1);
        const __amdgpu_buffer_rsrc_t ro = __builtin_amdgcn_make_buffer_rsrc((void*)p.out, 0, pass * (NR * MLOC * NLOC), 0x00020000);
        {
            const u64 give = !pass ? 0ull : wn ? 0x00ff00ff00ff00ffull : 0xff00ff00ff00ff00ull;
            const unsigned xo = (unsigned)((lane >> 4) * 8 + (lane & 7)) * 16u;
            int seen = 0;
            seen = -1;
            lds_wait_ge<1>(CTL_FREE + (lane & 3) * 4, pass * 2 * NH, seen, dead);
            asm volatile("s_mov_b64 exec, %[m]\n\t"
                "ds_write_b128 %[a], v[40:43] offset:0\n\tds_write_b128 %[a], v[44:47] offset:512\n\t"
                "ds_write_b128 %[a], v[48:51] offset:1024\n\tds_write_b128 %[a], v[52:55] offset:1536\n\t"
                "ds_write_b128 %[a], v[56:59] offset:2048\n\tds_write_b128 %[a], v[60:63] offset:2560\n\t"
                "ds_write_b128 %[a], v[64:67] offset:3072\n\tds_write_b128 %[a], v[68:71] offset:3584\n\t"
                "ds_write_b128 %[a], v[72:75] offset:4096\n\tds_write_b128 %[a], v[76:79] offset:4608\n\t"
                "ds_write_b128 %[a], v[80:83] offset:5120\n\tds_write_b128 %[a], v[84:87] offset:5632\n\t"
                "ds_write_b128 %[a], v[88:91] offset:6144\n\tds_write_b128 %[a], v[92:95] offset:6656\n\t"
                "ds_write_b128 %[a], v[96:99] offset:7168\n\tds_write_b128 %[a], v[100:103] offset:7680\n\t"
                "ds_write_b128 %[a], v[104:107] offset:8192\n\tds_write_b128 %[a], v[108:111] offset:8704\n\t"
                "ds_write_b128 %[a], v[112:115] offset:9216\n\tds_write_b128 %[a], v[116:119] offset:9728\n\t"
                "ds_write_b128 %[a], v[120:123] offset:10240\n\tds_write_b128 %[a], v[124:127] offset:10752\n\t"
                "ds_write_b128 %[a], v[128:131] offset:11264\n\tds_write_b128 %[a], v[132:135] offset:11776\n\t"
                "ds_write_b128 %[a], v[136:139] offset:12288\n\tds_write_b128 %[a], v[140:143] offset:12800\n\t"
                "ds_write_b128 %[a], v[144:147] offset:13312\n\tds_write_b128 %[a], v[148:151] offset:13824\n\t"
                "ds_write_b128 %[a], v[152:155] offset:14336\n\tds_write_b128 %[a], v[156:159] offset:14848\n\t"
                "ds_write_b128 %[a], v[160:163] offset:15360\n\tds_write_b128 %[a], v[164:167] offset:15872\n\t"
                "s_mov_b64 exec, -1"
                : "+{v[40:55]}"(acc[0]), "+{v[56:71]}"(acc[1]), "+{v[72:87]}"(acc[2]), "+{v[88:103]}"(acc[3]),
                  "+{v[104:119]}"(acc[4]), "+{v[120:135]}"(acc[5]), "+{v[136:151]}"(acc[6]), "+{v[152:167]}"(acc[7])
                : [a] "v"(LDS_A + wave * A_SLOT + xo), [m] "s"(give) : "memory");
            lds_st_u32(cm_ + (CTL_XCH - CTL_MISC) + wave * 4, (unsigned)pass);
            seen = -1;
            lds_wait_ge<1>(cm_ + (CTL_XCH - CTL_MISC) + (wave ^ 2) * 4, pass, seen, dead);
            asm volatile("s_mov_b64 exec, %[m]\n\t"
                "ds_read_b128 v[40:43], %[a] offset:0\n\tds_read_b128 v[44:47], %[a] offset:512\n\t"
                "ds_read_b128 v[48:51], %[a] offset:1024\n\tds_read_b128 v[52:55], %[a] offset:1536\n\t"
                "ds_read_b128 v[56:59], %[a] offset:2048\n\tds_read_b128 v[60:63], %[a] offset:2560\n\t"
                "ds_read_b128 v[64:67], %[a] offset:3072\n\tds_read_b128 v[68:71], %[a] offset:3584\n\t"
                "ds_read_b128 v[72:75], %[a] offset:4096\n\tds_read_b128 v[76:79], %[a] offset:4608\n\t"
                "ds_read_b128 v[80:83], %[a] offset:5120\n\tds_read_b128 v[84:87], %[a] offset:5632\n\t"
                "ds_read_b128 v[88:91], %[a] offset:6144\n\tds_read_b128 v[92:95], %[a] offset:6656\n\t"
                "ds_read_b128 v[96:99], %[a] offset:7168\n\tds_read_b128 v[100:103], %[a] offset:7680\n\t"
                "ds_read_b128 v[104:107], %[a] offset:8192\n\tds_read_b128 v[108:111], %[a] offset:8704\n\t"
                "ds_read_b128 v[112:115], %[a] offset:9216\n\tds_read_b128 v[116:119], %[a] offset:9728\n\t"
                "ds_read_b128 v[120:123], %[a] offset:10240\n\tds_read_b128 v[124:127], %[a] offset:10752\n\t"
                "ds_read_b128 v[128:131], %[a] offset:11264\n\tds_read_b128 v[132:135], %[a] offset:11776\n\t"
                "ds_read_b128 v[136:139], %[a] offset:12288\n\tds_read_b128 v[140:143], %[a] offset:12800\n\t"
                "ds_read_b128 v[144:147], %[a] offset:13312\n\tds_read_b128 v[148:151], %[a] offset:13824\n\t"
                "ds_read_b128 v[152:155], %[a] offset:14336\n\tds_read_b128 v[156:159], %[a] offset:14848\n\t"
                "ds_read_b128 v[160:163], %[a] offset:15360\n\tds_read_b128 v[164:167], %[a] offset:15872\n\t"
                "s_mov_b64 exec, -1\n\ts_waitcnt lgkmcnt(0)"
                : "+{v[40:55]}"(acc[0]), "+{v[56:71]}"(acc[1]), "+{v[72:87]}"(acc[2]), "+{v[88:103]}"(acc[3]),
                  "+{v[104:119]}"(acc[4]), "+{v[120:135]}"(acc[5]), "+{v[136:151]}"(acc[6]), "+{v[152:167]}"(acc[7])
                : [a] "v"(LDS_A + (wave ^ 2) * A_SLOT + xo), [m] "s"(give) : "memory");
        }
        const unsigned voff = (unsigned)(src * MLOC + mt * BM + wm * 128 + wn * 8 + (lane & 7)) * NLOC + (unsigned)(ct * BN + (lane & 8) * 8 + (lane >> 4) * 16);
        const int dead1 = dead;
        dead = 0;
        int iw = -1;
        lds_wait_ge<1>(cm_ + (CTL_GRDY - CTL_MISC), pass, iw, dead);
        iw = __builtin_amdgcn_readfirstlane(iw);
        if (__builtin_expect(dead | dead1 | (iw > 0x7f800000), 0)) {
            const unsigned nan4 = cm_ + (0x7f7f7f7fu - CTL_MISC);
#pragma unroll
            for (int i = 0; i < 8; ++i)
                __builtin_amdgcn_raw_buffer_store_b128((v4u){nan4, nan4, nan4, nan4}, ro, voff + (unsigned)i * 16u * NLOC, 0, CFG_OPOL);
        } else {
            const float inv = __int_as_float(iw - 1);
            const f32x2 inv2 = {inv, inv};
#pragma unroll
            for (int i = 0; i < 8; ++i) {
                f32x2 q[8];
#pragma unroll
                for (int c = 0; c < 8; ++c) q[c] = (f32x2){acc[i][2 * c], acc[i][2 * c + 1]} * inv2;
                v4u o;
#pragma unroll
                for (int rg = 0; rg < 4; ++rg) {
                    int w = __builtin_amdgcn_cvt_pk_fp8_f32(q[rg >> 1][rg & 1], q[2 + (rg >> 1)][rg & 1], 0, false);
                    w = __builtin_amdgcn_cvt_pk_fp8_f32(q[4 + (rg >> 1)][rg & 1], q[6 + (rg >> 1)][rg & 1], w, true);
                    o[rg] = (unsigned)w;
                }
                __builtin_amdgcn_raw_buffer_store_b128(o, ro, voff + (unsigned)i * 16u * NLOC, 0, CFG_OPOL);
                __builtin_amdgcn_sched_barrier(0);
            }
        }
        }
    } else if (wave == 4) {
        const __amdgpu_buffer_rsrc_t wrs = __builtin_amdgcn_make_buffer_rsrc((void*)p.w, 0, (int)((size_t)KDIM * NDIM * 4), 0x00020000);
        const __amdgpu_buffer_rsrc_t brs = __builtin_amdgcn_make_buffer_rsrc((void*)(myws + WS_WB), 0, 16 * NH * B_SLOT, 0x00020000);
        const unsigned wv = (unsigned)((lane >> 4) * 4) * (NDIM * 4) + (unsigned)(lane & 15) * 16u;
        unsigned c0_ = 0;
        if (rot > CFG_WE) BARRIER0();
#pragma nounroll
        for (int q = 0; q < 8; ++q) {
            const int ph = q * 16 + rot;
            if (ph > CFG_WE) UNTIL((ph - CFG_WE) * CFG_WSTEP, 4);
            f32x4 r[2][8];
#pragma unroll
            for (int h = 0; h < 2; ++h)
#pragma unroll
                for (int j = 0; j < 8; ++j)
                    r[h][j] = __builtin_bit_cast(f32x4, __builtin_amdgcn_raw_buffer_load_b128(wrs, wv + (unsigned)((j & 3) + (j >> 2) * 16) * (NDIM * 4) + h * 256u,
                        (unsigned)(ph * KH) * (NDIM * 4) + (unsigned)(p.rank * NLOC + (b >> 4) * BN) * 4u, 2));
            if (q == 0) c0_ = LOAD_CALL();
            if (ph <= CFG_WE) BARRIER0();
            __builtin_amdgcn_sched_barrier(0);
#pragma unroll
            for (int h = 0; h < 2; ++h)
#pragma unroll
                for (int c = 0; c < 4; ++c) {
                    v4u f;
                    f.x = cvt_bf16x2(r[h][0][c], r[h][1][c]); f.y = cvt_bf16x2(r[h][2][c], r[h][3][c]);
                    f.z = cvt_bf16x2(r[h][4][c], r[h][5][c]); f.w = cvt_bf16x2(r[h][6][c], r[h][7][c]);
                    __builtin_amdgcn_raw_buffer_store_b128(f, brs, vq + (unsigned)(h * 4096 + c * 1024), ((b >> 4) * NH + ph) * B_SLOT, 16);
                }
            asm volatile("s_waitcnt vmcnt(0)" ::: "memory");
            if (lane == 0) __hip_atomic_store((GAS unsigned*)(myws + WS_WF + (size_t)((b >> 4) * NH + ph) * WFS), c0_ + 1u, __ATOMIC_RELAXED, __HIP_MEMORY_SCOPE_AGENT);
        }
    } else if (wave == 7) {
        const int xcc = (int)__builtin_amdgcn_s_getreg((3 << 11) | 20) & 7;
        const int want = xcc * 2 + ((b >> 3) & 1);
        const unsigned po = lane >= 60 ? (unsigned)(WS_XP - WS_XF) + 128 - 960 + vq : lane == 4 * XDIR ? (unsigned)(WS_XP - WS_XF) : vq;
        BARRIER0();
        u64 k0 = 0;
        if (lane == 0) k0 = __hip_atomic_fetch_add((GAS u64*)(myws + WS_CLM + want * 256), 1ull, __ATOMIC_RELAXED, __HIP_MEMORY_SCOPE_AGENT);
        const unsigned c0_ = LOAD_CALL();
        v4u f0 = {~0u, ~0u, ~0u, ~0u};
        if (CFG_EARLY && (lane <= 4 * XDIR || lane >= 60))
            f0 = __builtin_amdgcn_raw_buffer_load_b128(__builtin_amdgcn_make_buffer_rsrc((void*)(myws + WS_XF + (size_t)(((p.srcmap >> (xcc * 4)) & 7) * 2 + (want & 1)) * 1024), 0,
                (int)(WS_XP - WS_XF) + 192, 0x00020000), po, 0, 17);
        __builtin_amdgcn_sched_barrier(0);
        const unsigned call = __builtin_amdgcn_readfirstlane(c0_) + 1u;
        int tile = xcc * 32 + __builtin_amdgcn_readfirstlane((int)k0) * 2 + (want & 1);
        bool early = CFG_EARLY;
        if ((unsigned)__builtin_amdgcn_readfirstlane((int)(k0 >> 32)) != call - 1u || (unsigned)__builtin_amdgcn_readfirstlane((int)k0) >= 16u) {
            tile = b; early = false;
#pragma nounroll
            for (int t = 0; t < 16; ++t) {
                const int g = ((want & ~1) + t) & 15;
                GAS u64* clm = (GAS u64*)(myws + WS_CLM + g * 256);
                unsigned k = 0;
                if (lane == 0) {
                    __hip_atomic_fetch_max(clm, (u64)(call - 1u) << 32, __ATOMIC_RELAXED, __HIP_MEMORY_SCOPE_AGENT);
                    k = (unsigned)__hip_atomic_fetch_add(clm, 1ull, __ATOMIC_RELAXED, __HIP_MEMORY_SCOPE_AGENT);
                }
                k = __builtin_amdgcn_readfirstlane(k);
                if (k < 16u) { tile = (g >> 1) * 32 + (int)k * 2 + (g & 1); break; }
            }
        }
        lds_st_u32(CTL_CALL, call);
        lds_st_u32(CTL_TILE, (unsigned)tile + 1u);
        TILE_VARS(tile);
        {
            const bool pub = src == p.rank && ct == 0;
            const __amdgpu_buffer_rsrc_t frs = __builtin_amdgcn_make_buffer_rsrc((void*)(myws + WS_XF + (size_t)(src * 2 + mt) * 1024), 0, (int)(WS_XP - WS_XF) + 192, 0x00020000);
            GAS u64* pw = (GAS u64*)(ws_of(p, lane & 7) + WS_XP + (size_t)(p.rank * 2 + mt) * 1024);
            const u64 t0 = rt();
            int i0 = 0, seen = 0;
            for (unsigned sp = 0; i0 < NH; ++sp) {
                asm volatile("" ::: "memory");
                v4u f = {call, call, call, call};
                if (pub) { if (lane >= (i0 >> 1)) f = __builtin_amdgcn_raw_buffer_load_b128(frs, vq, 0, 17); }
                else if (sp == 0 && early) f = f0;
                else if (lane == 4 * XDIR || (lane >= (i0 >> 3) * 4 && lane < 4 * XDIR) || (MS && !i0 && lane >= 60))
                    f = __builtin_amdgcn_raw_buffer_load_b128(frs, po, 0, 17);
                const u64 m = __builtin_amdgcn_ballot_w64(min(min(f.x, f.y), min(f.z, f.w)) < call) | (pub ? 0ull : 1ull << (4 * XDIR));
                int n = m ? (2 * __builtin_ctzll(m)) & ~7 : NH;
                if (!pub && (unsigned)__builtin_amdgcn_readlane((int)f.y, 4 * XDIR) == call) n = max(n, __builtin_amdgcn_readlane((int)f.x, 4 * XDIR));
                if (MS && !pub && !i0 && !(m >> 60)) n = max(n, MS);
                if (n > i0) {
                    i0 = n; lds_st_u32(CTL_XRDY, (unsigned)i0);
                    if (pub && n > 8 * XDIR && lane < NR) __hip_atomic_store(pw, ((u64)call << 32) | (unsigned)n, __ATOMIC_RELAXED, __HIP_MEMORY_SCOPE_SYSTEM);
                } else if ((sp & 63u) == 63u && timed_out(tmo, call, t0)) { lds_st_u32(CTL_BAD, 1u); lds_st_u32(CTL_XRDY, NH); break; }
                __builtin_amdgcn_s_sleep(CFG_PSL);
                if (pub) { if (i0 >= 8 * XDIR) __builtin_amdgcn_s_sleep(CFG_PUBSL); }
                else if (i0 < NH) lds_wait_ge<8>(CTL_FREE, 2 * (i0 - CFG_PAH), seen, dead);
            }
        }
        const __amdgpu_buffer_rsrc_t rs = __builtin_amdgcn_make_buffer_rsrc((void*)(myws + WS_GMAX + (size_t)(call & 1u) * 1024 + grp * 128), 0, 64, 0x00020000);
        const u64 t0 = rt();
        for (unsigned sp = 0;; ++sp) {
            asm volatile("" ::: "memory");
            const v2u g = __builtin_bit_cast(v2u, __builtin_amdgcn_raw_buffer_load_b64(rs, (lane & 7) * 8u, 0, 17));
            int gm = (int)g.x;
            bool ok = __all(g.y >= call);
            if (__builtin_expect((sp & 31u) == 31u, 0) && !ok && timed_out(tmo, call, t0)) { gm = 0x7fc00000; ok = true; }
            gm = max(gm, __builtin_amdgcn_update_dpp(0, gm, 0x124, 0xf, 0xf, false));
            gm = max(gm, __builtin_amdgcn_update_dpp(0, gm, 0x122, 0xf, 0xf, false));
            gm = max(gm, __builtin_amdgcn_update_dpp(0, gm, 0x121, 0xf, 0xf, false));
            const float amax = __int_as_float(gm);
            const float scale = amax / 448.0f;
            const float inv = amax > 0.f ? 1.0f / scale : 0.f;
            unsigned val = 1u + ((gm >= 0x7f800000 || !(inv <= 3.0e38f)) ? 0x7fc00000u : __float_as_uint(inv));
            asm volatile("" : "+v"(val));
            lds_st_u32(CTL_GRDY, ok ? val : 0u);
            if (ok) break;
            if (sp == 0) lds_wait_set<2>(CTL_DONE, dead);
        }
    } else {
        const int lw = wave - 5;
        const unsigned char* seq = myws + WS_SEQ + ((size_t)b * 2 + lw) * (NH * 4);
        __hip_atomic_store((GAS u64*)(seq + lane * 8), (u64)(unsigned)(2 * lane + 1) | ((u64)(lane < 63 ? (unsigned)(2 * lane + 2) : 0x1000u) << 32), __ATOMIC_RELAXED, __HIP_MEMORY_SCOPE_AGENT);
        BARRIER0();
        const int tile = lds_wait_set<1>(CTL_TILE, dead) - 1;
        const unsigned call = __builtin_amdgcn_readfirstlane(*(const volatile LAS unsigned*)(lds + CTL_CALL));
        TILE_VARS(tile);
        asm volatile("s_waitcnt vmcnt(0)" ::: "memory");
        const unsigned free_addr = CTL_FREE + (lane & 3) * 4;
        const unsigned v1 = vq + 4096, v2 = vq + 8192, v3 = vq + 12288, vz = 0;
#define DMA4(V, S, P) "global_load_lds_dwordx4 " V ", " S " offset:0" P "\n\tglobal_load_lds_dwordx4 " V ", " S " offset:1024" P "\n\t" \
                "global_load_lds_dwordx4 " V ", " S " offset:2048" P "\n\tglobal_load_lds_dwordx4 " V ", " S " offset:3072" P "\n\t"
        int seen = 0;
        if (lw == 0) {
            const unsigned char* xs = p.ws[src] + WS_XB + (size_t)(mt * NH) * A_SLOT;
            int rdy = 0;
            for (int i = -1; i < NH; ++i) {
                lds_wait_ge<1>(free_addr, 2 * (i - NA) + 2, seen, dead);
                lds_wait_ge<1>(CTL_XRDY, i + 1, rdy, dead);
                unsigned keep_;
#define XSTEP(XW) asm volatile(XW "s_cmp_lt_i32 %10, 0\n\ts_cselect_b64 exec, 0, -1\n\ts_mov_b32 %0, m0\n\ts_mov_b32 m0, %7\n\ts_nop 0\n\t" \
                    DMA4("%1", "%6", "") "s_add_u32 m0, m0, 0x1000\n\t" XW DMA4("%2", "%6", "") "s_add_u32 m0, m0, 0x1000\n\t" XW \
                    DMA4("%3", "%6", "") "s_add_u32 m0, m0, 0x1000\n\t" XW DMA4("%4", "%6", "") \
                    "s_mov_b32 m0, %9\n\ts_nop 0\n\tglobal_load_lds_dword %5, %8 sc1\n\ts_mov_b32 m0, %0\n\ts_mov_b64 exec, -1" \
                    : "=&s"(keep_) : "v"(vq), "v"(v1), "v"(v2), "v"(v3), "v"(vz), "s"(xs + (size_t)max(i, 0) * A_SLOT), \
                      "s"(LDS_A + (i & (NA - 1)) * A_SLOT), "s"(seq + max(i, 0) * 4), "s"(CTL_XFULL), "s"(i) : "memory", "scc")
                XSTEP("s_waitcnt vmcnt(" CFG_XCAP ")\n\t");
            }
        } else {
            const unsigned char* wsrc = myws + WS_WB + (size_t)(ct * NH) * B_SLOT;
            const __amdgpu_buffer_rsrc_t frs = __builtin_amdgcn_make_buffer_rsrc((void*)(myws + WS_WF + (size_t)(ct * NH) * WFS), 0, NH * WFS, 0x00020000);
            UNTIL(CFG_T0, 4);
            const u64 t0 = rt();
            int rdy = 0;
            int sb = 0, gi = ct % CFG_WG;
            for (int i = 0; i < NH; ++i) {
                for (unsigned sp = 0; rdy <= i; ++sp) {
                    asm volatile("" ::: "memory");
                    const bool no = lane >= CFG_WPW || __builtin_amdgcn_raw_buffer_load_b32(frs, (rdy + lane) * WFS, 0, 16) < call;
                    const u64 m = __builtin_amdgcn_ballot_w64(no);
                    rdy += m ? __builtin_ctzll(m) : 64;
                    if (rdy > i) break;
                    if ((sp & 63u) == 63u && timed_out(tmo, call, t0)) { dead = 1; rdy = NH; break; }
                    __builtin_amdgcn_s_sleep(CFG_PSL);
                }
                asm volatile("" ::: "memory");
                lds_wait_ge<2>(free_addr, 2 * (i - NB + (gi == 0 ? CFG_WG - 1 : 0)) + 1, seen, dead);
                if (++gi == CFG_WG) gi = 0;
                unsigned keep_;
                asm volatile("s_waitcnt vmcnt(52)\n\ts_mov_b32 %0, m0\n\ts_mov_b32 m0, %5\n\ts_nop 0\n\t"
                    DMA4("%1", "%4", " sc1") "s_add_u32 m0, m0, 0x1000\n\ts_nop 0\n\t" DMA4("%2", "%4", " sc1")
                    "s_mov_b32 m0, %7\n\ts_nop 0\n\tglobal_load_lds_dword %3, %6 sc1\n\ts_mov_b32 m0, %0"
                    : "=&s"(keep_) : "v"(vq), "v"(v1), "v"(vz), "s"(wsrc + (size_t)i * B_SLOT),
                      "s"(LDS_B + sb), "s"(seq + i * 4), "s"(CTL_WFULL) : "memory", "scc");
                sb += B_SLOT; if (sb == NB * B_SLOT) sb = 0;
            }
        }
        if (dead) lds_st_u32(CTL_BAD, 1u);
        asm volatile("s_waitcnt vmcnt(0)" ::: "memory");
        if (src == p.rank && (tile & 31) == 0) {
            lds_wait_set<2>(CTL_DONE, dead);
            const __amdgpu_buffer_rsrc_t rs = __builtin_amdgcn_make_buffer_rsrc((void*)(myws + WS_LMAX), 0, NWG * 32, 0x00020000);
            GAS u64* g = (GAS u64*)(ws_of(p, lane & 7) + WS_GMAX + (size_t)(call & 1u) * 1024 + (lane >> 3) * 128 + p.rank * 8);
            const u64 t0 = rt();
            int gm;
            for (unsigned sp = 0;; ++sp) {
                asm volatile("" ::: "memory");
                v4u t[8];
#pragma unroll
                for (int q = 0; q < 8; ++q) t[q] = __builtin_amdgcn_raw_buffer_load_b128(rs, vq, q * 1024, 16);
                unsigned lo = ~0u;
                gm = 0;
#pragma unroll
                for (int q = 0; q < 8; ++q) { gm = max(max(gm, (int)t[q].x), (int)t[q].z); lo = min(min(lo, t[q].y), t[q].w); }
                bool ok = __all(lo >= call);
                if (__builtin_expect((sp & 63u) == 63u, 0) && !ok && timed_out(tmo, call, t0)) { gm = 0x7fc00000; ok = true; }
                gm = wave_max(gm);
                asm volatile("" : "+v"(gm));
                if (ok) { __hip_atomic_store(g, ((u64)call << 32) | (unsigned)gm, __ATOMIC_RELAXED, __HIP_MEMORY_SCOPE_SYSTEM); break; }
            }
            if (lane < 16) __hip_atomic_store((GAS u64*)(myws + WS_CLM + lane * 256), (u64)call << 32, __ATOMIC_RELAXED, __HIP_MEMORY_SCOPE_AGENT);
            if (lane == 0) __hip_atomic_store((GAS unsigned*)(myws + WS_CALL), call, __ATOMIC_RELAXED, __HIP_MEMORY_SCOPE_AGENT);
        }
    }
}

extern "C" void kernel_launch(int rank, int nranks, ncclComm_t comm,
                              hipStream_t stream,
                              void* const* d_in, const int* in_sizes, int n_in,
                              void* d_out, int out_size,
                              const int* in_dtypes, int out_dtype,
                              void* const* d_out_all,
                              void* const* d_ws_all, size_t ws_bytes) {
    if (nranks != NR || (unsigned)rank >= (unsigned)NR || n_in < 2 || in_sizes[0] != MLOC * KDIM || in_sizes[1] != KDIM * NDIM ||
        out_size != NR * MLOC * NLOC || ws_bytes < WS_END) {
        fprintf(stderr, "kernel_launch: unexpected shapes\n");
        return;
    }
    HIP_CHECK(hipFuncSetAttribute((const void*)fused_kernel, hipFuncAttributeMaxDynamicSharedMemorySize, LDS_BYTES));
    Params p;
    p.x = (const float*)d_in[0];
    p.w = (const float*)d_in[1];
    p.out = (unsigned char*)d_out;
    for (int i = 0; i < NR; ++i) p.ws[i] = (unsigned char*)d_ws_all[i];
    p.myws = p.ws[rank];
    p.rank = rank;
    {
        const int cl[2][4] = {{0, 2, 3, 6}, {1, 4, 5, 7}}, own = (0xb2 >> rank) & 1, slot[3] = {0, 1, 3};
        int m[NR], k = 0;
        for (int i = 0; i < 4; ++i) { m[4 + i] = cl[own ^ 1][i]; if (cl[own][i] != rank) m[slot[k++]] = cl[own][i]; }
        m[2] = rank;
        p.srcmap = 0;
        for (int i = 0; i < NR; ++i) p.srcmap |= (unsigned)m[i] << (4 * i);
    }
    hipLaunchKernelGGL(fused_kernel, dim3(NWG), dim3(NTHREADS), LDS_BYTES, stream, p);
    HIP_CHECK(hipGetLastError());
}
```
